```python
import numpy as np

kernel_name = "gpu_a2a_gemm_m4096_k4096_n16384_bf16_relu"


ABSMAX_RTOL = 0.008

COMPUTE_DTYPE = "bfloat16"
EPILOGUE = "relu"
IN_QUANT = None
OUT_QUANT = None
IN_DTYPES = {"x": "bfloat16", "w_mat": "bfloat16"}
WORLD_SIZE = 8

def _to_bf16(x):
    u = np.ascontiguousarray(x, dtype=np.float32).view(np.uint32)
    return ((u + 0x7FFF + ((u >> 16) & 1)) & 0xFFFF0000).view(np.float32)

def setup_inputs():

    x = _to_bf16(np.random.randn(4096, 4096).astype(np.float32))
    w_mat = _to_bf16((np.random.randn(4096, 16384) / np.sqrt(4096)).astype(np.float32))
    return {"x": x, "w_mat": w_mat}

def _epilogue(y):
    return np.maximum(y, 0.0).astype(np.float32)

def reference(x, w_mat):
    y = (x @ w_mat).astype(np.float32)
    return _epilogue(y)
```

```cpp
#include <hip/hip_runtime.h>
#include <rccl/rccl.h>
#include <cstdio>

#define HIP_CHECK(expr) do { hipError_t _e = (expr); if (_e != hipSuccess) { \
    fprintf(stderr, "HIP error %s at %s:%d\n", hipGetErrorString(_e), __FILE__, __LINE__); \
    return; } } while (0)

namespace a2ag {

typedef __bf16 bf16x8 __attribute__((ext_vector_type(8)));
typedef __bf16 bf16x2 __attribute__((ext_vector_type(2)));
typedef short s16x4 __attribute__((ext_vector_type(4)));
typedef short s16x8 __attribute__((ext_vector_type(8)));
typedef float f32x2 __attribute__((ext_vector_type(2)));
typedef float f32x4 __attribute__((ext_vector_type(4)));
typedef unsigned u32x4 __attribute__((ext_vector_type(4)));
#define LDS __attribute__((address_space(3)))
#define DI __device__ __forceinline__

constexpr int NR = 8;
constexpr int MLOC = 512;
constexpr int KSH = 512;
constexpr int KTOT = 4096, NTOT = 16384;
constexpr int NSTEP = KTOT / 64;
constexpr int NWG = 256;
constexpr unsigned A_SLOT = 32768, B_SLOT = 16384;
constexpr unsigned B_BASE = 3 * A_SLOT;
constexpr unsigned LDS_BYTES = B_BASE + 4 * B_SLOT;
constexpr unsigned X_BYTES = 4096u * KSH * 2u;
constexpr unsigned W_BYTES = (unsigned)KTOT * NTOT * 2u;
constexpr unsigned OUT_BYTES = (unsigned)MLOC * NTOT * 2u;

constexpr unsigned WS_SRC = 512u * 1024u;
constexpr unsigned WS_PAR = NR * WS_SRC;
constexpr unsigned WS_CNT = 2 * WS_PAR;
constexpr unsigned WS_CALL = WS_CNT + NR * 8 * 2 * 128;
constexpr unsigned WS_TMO = WS_CALL + NWG * 128;
constexpr unsigned WS_END = WS_TMO + 128;
constexpr unsigned PUSH_WGS = 7 * 32;
constexpr unsigned CNT_PER_CALL = 16;
constexpr unsigned long long DEADLINE_TICKS = 50000000ull;

#define PUSH_D0 3
#define PUSH_D 5

struct Params {
    const void* x; const void* w; void* out;
    char* ws[NR];
    int rank; int pad;
};

#define BAR() do { asm volatile("" ::: "memory"); __builtin_amdgcn_s_barrier(); asm volatile("" ::: "memory"); } while (0)
#define WAITV(n) asm volatile("s_waitcnt vmcnt(" #n ")" ::: "memory")

DI unsigned rfl(unsigned v) { return __builtin_amdgcn_readfirstlane(v); }

DI void dma_plain(unsigned lds, unsigned voff, __amdgpu_buffer_rsrc_t srd, unsigned soff) {
    asm volatile("s_mov_b32 m0, %0\n\ts_nop 0\n\tbuffer_load_dwordx4 %1, %2, %3 offen lds"
                 :: "s"(lds), "v"(voff), "s"(srd), "s"(soff) : "memory");
}
DI void dma_sys4(unsigned lds, unsigned voff, __amdgpu_buffer_rsrc_t srd, unsigned soff) {
    asm volatile("s_mov_b32 m0, %0\n\ts_nop 0\n\t"
                 "buffer_load_dwordx4 %1, %2, %3 offen sc0 sc1 lds\n\t"
                 "buffer_load_dwordx4 %1, %2, %3 offen offset:1024 sc0 sc1 lds\n\t"
                 "buffer_load_dwordx4 %1, %2, %3 offen offset:2048 sc0 sc1 lds\n\t"
                 "buffer_load_dwordx4 %1, %2, %3 offen offset:3072 sc0 sc1 lds"
                 :: "s"(lds), "v"(voff), "s"(srd), "s"(soff) : "memory");
}

DI void step_map(int s, int& j, int& kc) {
    if (s < 8) { j = 0; kc = s; }
    else { const int u = s - 8; kc = (u * 147) >> 10; j = 1 + (u - 7 * kc); }
}

DI unsigned pack_relu(float a, float b) {
    f32x2 v = {__builtin_elementwise_maximum(a, 0.f), __builtin_elementwise_maximum(b, 0.f)};
    bf16x2 r = __builtin_convertvector(v, bf16x2);
    return __builtin_bit_cast(unsigned, r);
}

__global__ __launch_bounds__(512, 2) void a2a_gemm(Params p) {
    extern __shared__ __attribute__((aligned(16))) char smem[];
    const unsigned ldsb = (unsigned)(size_t)(LDS char*)smem;
    const unsigned tid = threadIdx.x, lane = tid & 63;
    const unsigned wid = rfl(tid >> 6);
    const unsigned b = blockIdx.x;
    const unsigned nt = (b & 7) * 16 + (b >> 4), mh = (b >> 3) & 1;
    const unsigned rank = (unsigned)p.rank & (NR - 1);
    char* const myws = p.ws[rank];

    if (wid < 4) {
        const unsigned wm = wid >> 1, wn = wid & 1;
        const unsigned fr = lane & 15, fq = lane >> 4;
        const unsigned a0 = ldsb + 128u * (wm * 128 + fr) + 16u * (fq ^ (fr >> 1));
        const unsigned a1 = ldsb + 128u * (wm * 128 + fr) + 16u * ((fq + 4) ^ (fr >> 1));
        const unsigned q = (lane >> 2) & 3, pp = lane & 3;
        const unsigned fB = (q & 2) | ((fq & 1) << 2);
        unsigned bb[4];
#pragma unroll
        for (int c = 0; c < 4; ++c)
            bb[c] = ldsb + B_BASE + wn * 8192u + 128u * (8 * fq + q) + 16u * ((2 * c + (pp >> 1)) ^ fB) + 8u * (pp & 1);

        f32x4 acc[8][4];
#pragma unroll
        for (int m = 0; m < 8; ++m)
#pragma unroll
            for (int c = 0; c < 4; ++c) acc[m][c] = (f32x4){0.f, 0.f, 0.f, 0.f};
        bf16x8 X0[8], X1[8], W0[4], W1[4];

#define RD_X(dst, base) do { _Pragma("unroll") for (int m = 0; m < 8; ++m) dst[m] = *(const LDS bf16x8*)((base) + 2048u * m); } while (0)
#define RD_W(dst, boff, ks) do { _Pragma("unroll") for (int c = 0; c < 4; ++c) { \
            const s16x4 lo = __builtin_amdgcn_ds_read_tr16_b64_v4i16((LDS s16x4*)(bb[c] + (boff) + 4096u * (ks))); \
            const s16x4 hi = __builtin_amdgcn_ds_read_tr16_b64_v4i16((LDS s16x4*)(bb[c] + (boff) + 4096u * (ks) + 512u)); \
            dst[c] = __builtin_bit_cast(bf16x8, __builtin_shufflevector(lo, hi, 0, 1, 2, 3, 4, 5, 6, 7)); } } while (0)
#define MMA(X, W) do { _Pragma("unroll") for (int m = 0; m < 8; ++m) _Pragma("unroll") for (int c = 0; c < 4; ++c) \
            acc[m][c] = __builtin_amdgcn_mfma_f32_16x16x32_bf16(W[c], X[m], acc[m][c], 0, 0, 0); } while (0)
#define SCHED_SUB() do { _Pragma("unroll") for (int i = 0; i < 16; ++i) { \
            __builtin_amdgcn_sched_group_barrier(0x008, 1, 0); __builtin_amdgcn_sched_group_barrier(0x100, 1, 0); } \
            __builtin_amdgcn_sched_group_barrier(0x008, 16, 0); } while (0)

        BAR();
        RD_W(W0, 0u, 0); RD_X(X0, a0);
        unsigned aoff = 0, boff = 0;
        for (int k = 0; k < NSTEP; ++k) {
            __builtin_amdgcn_sched_barrier(0);
            RD_W(W1, boff, 1); RD_X(X1, a1 + aoff);
            MMA(X0, W0);
            SCHED_SUB();
            __builtin_amdgcn_sched_barrier(0);
            __builtin_amdgcn_s_waitcnt(0xC07F);
            BAR();
            __builtin_amdgcn_sched_barrier(0);
            aoff = (aoff == 2 * A_SLOT) ? 0u : aoff + A_SLOT;
            boff = (boff + B_SLOT) & (4 * B_SLOT - 1);
            RD_W(W0, boff, 0); RD_X(X0, a0 + aoff);
            MMA(X1, W1);
            SCHED_SUB();
        }
        __builtin_amdgcn_sched_barrier(0);
        const unsigned call = __hip_atomic_load((unsigned*)(myws + WS_CALL + b * 128), __ATOMIC_RELAXED, __HIP_MEMORY_SCOPE_AGENT) + 1u;
        const bool bad = __hip_atomic_load((unsigned*)(myws + WS_TMO), __ATOMIC_RELAXED, __HIP_MEMORY_SCOPE_AGENT) == call;
        char* orow = (char*)p.out + (size_t)(mh * 256 + wm * 128 + fr) * (NTOT * 2) + (nt * 128 + wn * 64 + 16 * (fq & 1) + 8 * (fq >> 1)) * 2;
#pragma unroll
        for (int m = 0; m < 8; ++m)
#pragma unroll
            for (int cp = 0; cp < 2; ++cp) {
                const f32x4 u = acc[m][2 * cp], v = acc[m][2 * cp + 1];
                const unsigned u0 = pack_relu(u[0], u[1]), u1 = pack_relu(u[2], u[3]);
                const unsigned v0 = pack_relu(v[0], v[1]), v1 = pack_relu(v[2], v[3]);
                const auto s0 = __builtin_amdgcn_permlane16_swap(u0, v0, false, false);
                const auto s1 = __builtin_amdgcn_permlane16_swap(u1, v1, false, false);
                u32x4 o = {s0[0], s1[0], s0[1], s1[1]};
                if (bad) o = (u32x4){0x7fc07fc0u, 0x7fc07fc0u, 0x7fc07fc0u, 0x7fc07fc0u};
                *(u32x4*)(orow + (size_t)m * 16 * (NTOT * 2) + cp * 64) = o;
            }
        BAR();
    } else if (wid < 6) {
        const unsigned aw = wid - 4;
        unsigned cv;
        asm volatile("global_load_dword %0, %1, off sc1" : "=v"(cv) : "v"(myws + WS_CALL + b * 128) : "memory");
        const __amdgpu_buffer_rsrc_t srdX = __builtin_amdgcn_make_buffer_rsrc((void*)p.x, (short)0, (int)X_BYTES, 0x00020000);
        const __amdgpu_buffer_rsrc_t srdR = __builtin_amdgcn_make_buffer_rsrc((void*)myws, (short)0, (int)WS_CNT, 0x00020000);
        unsigned vL[2];
#pragma unroll
        for (int par = 0; par < 2; ++par) vL[par] = (lane >> 3) * 1024u + (((lane & 7) ^ (4 * par + (lane >> 4))) * 16u);
        const unsigned vR = lane * 16u;
        const unsigned sL = (rank * 512u + mh * 256u + aw * 128u) * 1024u;
        unsigned sR = 0;
        const unsigned ldw = ldsb + aw * 16384u;
        unsigned* const tmo = (unsigned*)(myws + WS_TMO);

        auto issue = [&](int s, unsigned slot) {
            int j, kc; step_map(s, j, kc);
            if (j == 0) {
#pragma unroll
                for (int i = 0; i < 16; ++i) dma_plain(ldw + slot + 1024u * i, vL[i & 1], srdX, sL + 8192u * i + 128u * kc);
            } else {
                const unsigned src = (rank + j) & (NR - 1);
#pragma unroll
                for (int i = 0; i < 4; ++i) dma_sys4(ldw + slot + 4096u * i, vR, srdR, sR + src * WS_SRC + 65536u * kc + 4096u * i);
            }
        };
        auto cnt_ptr = [&](int s) -> const unsigned* {
            int j, kc; step_map(s, j, kc);
            const unsigned src = (rank + j) & (NR - 1);
            return (const unsigned*)(myws + WS_CNT + ((src * 8u + kc) * 2u + mh) * 128u);
        };
        issue(0, 0); issue(1, A_SLOT); issue(2, 2 * A_SLOT);
        asm volatile("s_waitcnt vmcnt(32)" : "+v"(cv) :: "memory");
        const unsigned call = rfl(cv) + 1u;
        const unsigned target = call * CNT_PER_CALL;
        sR = (call & 1u) * WS_PAR + mh * 32768u + aw * 16384u;
        BAR();
        WAITV(16);
        unsigned slot = 0, seen = 0;
        for (int k = 0; k < NSTEP; ++k) {
            BAR();
            const int s = k + 3;
            if (s < NSTEP) {
                if (s >= 8 && seen < target) {
                    const unsigned* c = cnt_ptr(s);
                    const unsigned long long t0 = __builtin_amdgcn_s_memrealtime();
                    for (;;) {
                        unsigned v, t;
                        asm volatile("global_load_dword %0, %2, off sc0 sc1\n\tglobal_load_dword %1, %3, off sc1\n\ts_waitcnt vmcnt(0)"
                                     : "=&v"(v), "=&v"(t) : "v"(c), "v"(tmo) : "memory");
                        if (rfl(v) >= target) break;
                        if (rfl(t) == call || __builtin_amdgcn_s_memrealtime() - t0 > DEADLINE_TICKS) {
                            asm volatile("global_store_dword %0, %1, off sc1\n\ts_waitcnt vmcnt(0)" :: "v"(tmo), "v"(call) : "memory");
                            break;
                        }
                        __builtin_amdgcn_s_sleep(2);
                    }
                }
                unsigned pv = 0;
                const bool pre = (s + 1 >= 8) && (s + 1 < NSTEP);
                if (pre) { const unsigned* c = cnt_ptr(s + 1); asm volatile("global_load_dword %0, %1, off sc0 sc1" : "=v"(pv) : "v"(c) : "memory"); }
                issue(s, slot);
                asm volatile("s_waitcnt vmcnt(16)" : "+v"(pv) :: "memory");
                seen = rfl(pv);
            } else {
                WAITV(0);
            }
            slot = (slot == 2 * A_SLOT) ? 0u : slot + A_SLOT;
        }
        BAR();
    } else if (wid == 6) {
        const __amdgpu_buffer_rsrc_t srdW = __builtin_amdgcn_make_buffer_rsrc((void*)p.w, (short)0, (int)W_BYTES, 0x00020000);
        unsigned vW[2];
#pragma unroll
        for (int par = 0; par < 2; ++par) vW[par] = (lane >> 3) * (NTOT * 2u) + (((lane & 7) ^ (((lane >> 3) & 2) | (par << 2))) * 16u);
        const unsigned ldw = ldsb + B_BASE;
        auto issue = [&](int s, unsigned slot) {
            int j, kc; step_map(s, j, kc);
            const unsigned k0 = ((rank + j) & (NR - 1)) * KSH + kc * 64u;
            const unsigned so = k0 * (NTOT * 2u) + nt * 256u;
#pragma unroll
            for (int i = 0; i < 16; ++i)
                dma_plain(ldw + slot + 1024u * i, vW[i & 1], srdW, so + (i & 7) * 8u * (NTOT * 2u) + (i >> 3) * 128u);
        };
        issue(0, 0); issue(1, B_SLOT); issue(2, 2 * B_SLOT);
        WAITV(32);
        issue(3, 3 * B_SLOT);
        BAR();
        WAITV(32);
        unsigned slot = 0;
        for (int k = 0; k < NSTEP; ++k) {
            BAR();
            if (k + 4 < NSTEP) { issue(k + 4, slot); WAITV(32); }
            else if (k + 4 == NSTEP) { WAITV(16); }
            else { WAITV(0); }
            slot = (slot + B_SLOT) & (4 * B_SLOT - 1);
        }
        BAR();
    } else {
        unsigned* const callp = (unsigned*)(myws + WS_CALL + b * 128);
        unsigned cv;
        asm volatile("global_load_dword %0, %1, off sc1" : "=v"(cv) : "v"(callp) : "memory");
        int nb = NSTEP + 1;
        if (b < PUSH_WGS) {
            const unsigned rg = b / 7u, dst = (rank + 1u + (b - rg * 7u)) & (NR - 1);
            const char* sp[2];
#pragma unroll
            for (int h = 0; h < 2; ++h)
                sp[h] = (const char*)p.x + (size_t)(dst * 512u + rg * 16u + h * 8u + (lane >> 3)) * 1024u + (((lane & 7) ^ (4 * h + (lane >> 4))) * 16u);
            unsigned* cp = (unsigned*)(p.ws[dst] + WS_CNT + (rank * 16u + (rg >> 4)) * 128u);
            u32x4 d[8][2];
#pragma unroll
            for (int kc = 0; kc < 8; ++kc)
#pragma unroll
                for (int h = 0; h < 2; ++h) d[kc][h] = __builtin_nontemporal_load((const u32x4*)(sp[h] + kc * 128));
            asm volatile("s_waitcnt vmcnt(16)" : "+v"(cv) :: "memory");
            char* dp = p.ws[dst] + ((rfl(cv) + 1u) & 1u) * WS_PAR + rank * WS_SRC + rg * 2048u + lane * 16u;
#pragma unroll
            for (int kc = 0; kc < 8; ++kc) {
                asm volatile("global_store_dwordx4 %0, %1, off sc0 sc1\n\tglobal_store_dwordx4 %0, %2, off offset:1024 sc0 sc1\n\ts_nop 1"
                             :: "v"(dp + kc * 65536u), "v"(d[kc][0]), "v"(d[kc][1]) : "memory");
                const int nwait = (kc == 0) ? PUSH_D0 : PUSH_D;
                for (int i = 0; i < nwait; ++i) BAR();
                nb -= nwait;
                WAITV(0);
                if (lane == 0) __hip_atomic_fetch_add(cp + kc * 64, 1u, __ATOMIC_RELAXED, __HIP_MEMORY_SCOPE_SYSTEM);
            }
        }
        for (int i = 0; i < nb; ++i) BAR();
        BAR();
        asm volatile("s_waitcnt vmcnt(0)" : "+v"(cv) :: "memory");
        if (lane == 0) __hip_atomic_store(callp, cv + 1u, __ATOMIC_RELAXED, __HIP_MEMORY_SCOPE_AGENT);
    }
}

}

extern "C" void kernel_launch(int rank, int nranks, ncclComm_t comm,
                              hipStream_t stream,
                              void* const* d_in, const int* in_sizes, int n_in,
                              void* d_out, int out_size,
                              const int* in_dtypes, int out_dtype,
                              void* const* d_out_all,
                              void* const* d_ws_all, size_t ws_bytes) {
    using namespace a2ag;
    if (nranks != NR || (unsigned)rank >= (unsigned)NR || n_in < 2 || ws_bytes < WS_END ||
        in_sizes[0] != 4096 * KSH || in_sizes[1] != KTOT * NTOT || out_size != MLOC * NTOT) {
        fprintf(stderr, "kernel_launch: unexpected shapes\n");
        return;
    }
    HIP_CHECK(hipFuncSetAttribute((const void*)a2a_gemm, hipFuncAttributeMaxDynamicSharedMemorySize, (int)LDS_BYTES));
    Params p;
    p.x = d_in[0]; p.w = d_in[1]; p.out = d_out;
    for (int i = 0; i < NR; ++i) p.ws[i] = (char*)d_ws_all[i];
    p.rank = rank; p.pad = 0;
    hipLaunchKernelGGL(a2a_gemm, dim3(NWG), dim3(512), LDS_BYTES, stream, p);
    HIP_CHECK(hipGetLastError());
}
```

```cpp
#include <hip/hip_runtime.h>
#include <rccl/rccl.h>
#include <cstdio>

#define HIP_CHECK(expr) do { hipError_t _e = (expr); if (_e != hipSuccess) { \
    fprintf(stderr, "HIP error %s at %s:%d\n", hipGetErrorString(_e), __FILE__, __LINE__); \
    return; } } while (0)

namespace a2ag {

typedef __bf16 bf16x8 __attribute__((ext_vector_type(8)));
typedef __bf16 bf16x2 __attribute__((ext_vector_type(2)));
typedef short s16x4 __attribute__((ext_vector_type(4)));
typedef float f32x2 __attribute__((ext_vector_type(2)));
typedef float f32x4 __attribute__((ext_vector_type(4)));
typedef unsigned u32x4 __attribute__((ext_vector_type(4)));
#define LDS __attribute__((address_space(3)))
#define DI __device__ __forceinline__

constexpr int NR = 8;
constexpr int MLOC = 512;
constexpr int KSH = 512;
constexpr int KTOT = 4096, NTOT = 16384;
constexpr int NSTEP = KTOT / 64;
constexpr int NWG = 256;
constexpr unsigned B_SLOT = 32768, A_SLOT = 16384;
constexpr unsigned A_BASE = 3 * B_SLOT;
constexpr unsigned P_BASE = A_BASE + 3 * A_SLOT;
constexpr unsigned LDS_BYTES = P_BASE + 16384;
constexpr unsigned X_BYTES = 4096u * KSH * 2u;
constexpr unsigned W_BYTES = (unsigned)KTOT * NTOT * 2u;
constexpr unsigned WROW = NTOT * 2u;

constexpr unsigned WS_SRC = 512u * 1024u;
constexpr unsigned WS_PAR = NR * WS_SRC;
constexpr unsigned WS_CNT = 2 * WS_PAR;
constexpr unsigned WS_CALL = WS_CNT + NR * 2 * 128;
constexpr unsigned WS_TMO = WS_CALL + NWG * 128;
constexpr unsigned WS_END = WS_TMO + 128;
constexpr unsigned PUSH_WGS = 7 * 32;
constexpr unsigned CNT_PER_CALL = 16;
constexpr unsigned long long DEADLINE_TICKS = 50000000ull;

#define PUSH_D0 3
#define PUSH_D 5
#define PUSH_GAP 3
#define KORD 2
#define POLL_AHEAD 3

#define ST_DECL
#define ST_BEG()
#define ST_END(acc)
#define ST_FIRST()
#define ST_OUT(x6)

struct Params {
    const void* x; const void* w; void* out; char* myws;
    char* ws[NR];
    int rank; int pad;
};

#define BAR() do { asm volatile("" ::: "memory"); __builtin_amdgcn_s_barrier(); asm volatile("" ::: "memory"); } while (0)
#define WAITV(n) asm volatile("s_waitcnt vmcnt(" #n ")" ::: "memory")
#define SB0() __builtin_amdgcn_sched_barrier(0)

DI unsigned rfl(unsigned v) { return __builtin_amdgcn_readfirstlane(v); }

#define DMA4(suffix, lds, v, srd, soff) asm volatile("s_mov_b32 m0, %0\n\ts_nop 0\n\t" \
        "buffer_load_dwordx4 %1, %5, %6 offen " suffix " lds\n\t" \
        "buffer_load_dwordx4 %2, %5, %6 offen offset:1024 " suffix " lds\n\t" \
        "buffer_load_dwordx4 %3, %5, %6 offen offset:2048 " suffix " lds\n\t" \
        "buffer_load_dwordx4 %4, %5, %6 offen offset:3072 " suffix " lds" \
        :: "s"(lds), "v"((v)[0]), "v"((v)[1]), "v"((v)[2]), "v"((v)[3]), "s"(srd), "s"(soff) : "memory")

DI void step_map(unsigned t, unsigned& j, unsigned& kc) {
    if (t < 8) { j = 0; kc = t; }
    else { const unsigned u = t - 8; kc = (u * 147u) >> 10; j = 1 + (u - 7 * kc); }
}

DI unsigned stage_map(unsigned s, unsigned q) {
    const unsigned o = (q + 2u) & 3u, v = (s - o) & 3u, s0 = s - v;
    if (s < o || s0 + 3u >= NSTEP || (s0 < 8u && s0 + 2u >= 8u)) return s;
    return v == 0 ? s + 2u : (v == 3 ? s : s - 1u);
}

DI unsigned pack_relu(float a, float b) {
    f32x2 v = {__builtin_elementwise_maximum(a, 0.f), __builtin_elementwise_maximum(b, 0.f)};
    bf16x2 r = __builtin_convertvector(v, bf16x2);
    return __builtin_bit_cast(unsigned, r);
}

__global__ __launch_bounds__(512, 2) void a2a_gemm(Params p) {
    extern __shared__ __attribute__((aligned(16))) char smem[];
    const unsigned ldsb = (unsigned)(size_t)(LDS char*)smem;
    const unsigned tid = threadIdx.x, lane = tid & 63;
    const unsigned wid = rfl(tid >> 6);
    const unsigned b = blockIdx.x;
    const unsigned mq = (b >> 3) & 3, mh = mq >> 1;
    const unsigned colb = (b & 7) * 4096u + (b >> 5) * 512u;
    const unsigned rank = (unsigned)p.rank & (NR - 1);
    const unsigned rg = b / 7u, dst = (rank + 1u + (b - rg * 7u)) & (NR - 1);
    char* const myws = p.myws;
    unsigned* const callp = (unsigned*)(myws + WS_CALL + b * 128);
    unsigned* const tmo = (unsigned*)(myws + WS_TMO);
#define GET_CALL() (rfl(*(const volatile unsigned __attribute__((address_space(4)))*)(size_t)callp) + 1u)
    ST_DECL;

    if (wid >= 4) {
        const unsigned lw = wid - 4;
        const __amdgpu_buffer_rsrc_t srdW = __builtin_amdgcn_make_buffer_rsrc((void*)p.w, (short)0, (int)W_BYTES, 0x00020000);
        const __amdgpu_buffer_rsrc_t srdX = __builtin_amdgcn_make_buffer_rsrc((void*)p.x, (short)0, (int)X_BYTES, 0x00020000);
        const __amdgpu_buffer_rsrc_t srdR = __builtin_amdgcn_make_buffer_rsrc((void*)myws, (short)0, (int)WS_CNT, 0x00020000);
        unsigned vW[4], vL[4], vR[4];
#pragma unroll
        for (int j = 0; j < 4; ++j) {
            vW[j] = (lane >> 3) * WROW + (((lane & 7) ^ (((lane >> 3) & 2) | ((j & 1) << 2))) * 16u) + j * (8u * WROW - 1024u);
            vL[j] = (lane >> 3) * 1024u + (((lane & 7) ^ (4 * (j & 1) + (lane >> 4))) * 16u) + j * (8192u - 1024u);
            vR[j] = lane * 16u;
        }
        const unsigned sW = colb + lw * 128u;
        const unsigned lW = ldsb + lw * 8192u;
        const unsigned sL = (rank * 512u + mq * 128u + lw * 32u) * 1024u;
        const unsigned lA = ldsb + A_BASE + lw * 4096u;
        unsigned sR = 0;
        auto issue = [&](unsigned s, unsigned slot) {
            unsigned j, kc; step_map(stage_map(s, mq), j, kc);
            const unsigned src = (rank + j) & (NR - 1);
            const unsigned so = sW + (src * KSH + kc * 64u) * WROW;
            {
            DMA4("", lW + slot * B_SLOT, vW, srdW, so);
            DMA4("", lW + slot * B_SLOT + 4096u, vW, srdW, so + 32u * WROW);
            }
            {
            if (j == 0) DMA4("", lA + slot * A_SLOT, vL, srdX, sL + 128u * kc);
            else        DMA4("sc0 sc1", lA + slot * A_SLOT, vR, srdR, sR + src * WS_SRC + 65536u * kc);
            }
        };
        issue(0, 0); issue(1, 1);
        WAITV(12); BAR();
        issue(2, 2);
        {
            const unsigned so = (dst * 512u + (rg & 31u) * 16u) * 1024u + lw * 256u;
#pragma unroll
            for (int j = 0; j < 4; ++j)
                asm volatile("s_mov_b32 m0, %0\n\ts_nop 0\n\tbuffer_load_dwordx4 %1, %2, %3 offen nt lds"
                             :: "s"(ldsb + P_BASE + lw * 4096u + j * 1024u), "v"(vL[j & 1] + (j & 1) * 1024u), "s"(srdX), "s"(so + (j >> 1) * 128u) : "memory");
        }
        if (lw == 3) {
            unsigned dummy;
            const unsigned vp = (rank * 512u + mq * 128u + (b >> 5) * 16u + (lane & 15)) * 1024u + (4u + (lane >> 4)) * 128u;
            asm volatile("buffer_load_dword %0, %1, %2, 0 offen" : "=v"(dummy) : "v"(vp), "s"(srdX) : "memory");
        }
        const unsigned call = GET_CALL();
        sR = (call & 1u) * WS_PAR + mq * 16384u + lw * 4096u;
        WAITV(16);
        const unsigned target = call * CNT_PER_CALL;
        unsigned long long ready = 0xffull;
        const unsigned* cptr;
        { unsigned j, kc; step_map(stage_map(8u + (lane < 56 ? lane : 55u), mq), j, kc);
          cptr = (const unsigned*)(myws + WS_CNT + (((rank + j) & (NR - 1)) * 2u + mh) * 128u + kc * 4u);
          ready |= __ballot(j == 0) << 8; }
        unsigned slot = 0;
        for (unsigned k = 0; k < NSTEP; ++k) {
            ST_BEG();
            BAR();
            ST_END(st_bar);
            unsigned pv = 0;
            const bool poll = (lw == 0) && k + 4 < NSTEP && (((~ready) >> (k + 4)) & ((2ull << POLL_AHEAD) - 1ull)) != 0;
            if (poll) asm volatile("global_load_dword %0, %1, off sc0 sc1" : "=v"(pv) : "v"(cptr) : "memory");
            if (k + 3 < NSTEP) issue(k + 3, slot);
            ST_BEG();
            if (k + 3 < NSTEP) WAITV(12); else WAITV(0);
            ST_END(st_vm);
            if (poll) {
                asm volatile("" : "+v"(pv));
                ready |= __ballot(pv >= target) << 8;
                if (!((ready >> (k + 4)) & 1ull)) {
                    const unsigned long long t0 = __builtin_amdgcn_s_memrealtime();
                    for (;;) {
                        unsigned v, t;
                        asm volatile("global_load_dword %0, %2, off sc0 sc1\n\tglobal_load_dword %1, %3, off sc1\n\ts_waitcnt vmcnt(0)"
                                     : "=&v"(v), "=&v"(t) : "v"(cptr), "v"(tmo) : "memory");
                        ready |= __ballot(v >= target) << 8;
                        if ((ready >> (k + 4)) & 1ull) break;
                        if (rfl(t) == call || __builtin_amdgcn_s_memrealtime() - t0 > DEADLINE_TICKS) {
                            asm volatile("global_store_dword %0, %1, off sc1\n\ts_waitcnt vmcnt(0)" :: "v"(tmo), "v"(call) : "memory");
                            ready = ~0ull;
                            break;
                        }
                        __builtin_amdgcn_s_sleep(2);
                    }
                }
            }
            slot = (slot == 2) ? 0u : slot + 1;
        }
        BAR();
        if (lw == 0 && lane == 0) __hip_atomic_store(callp, call, __ATOMIC_RELAXED, __HIP_MEMORY_SCOPE_AGENT);
        ST_OUT(st_pollk);
    } else {
        const unsigned fr = lane & 15, fq = lane >> 4;
        const unsigned vd = lane * 16u;
        const unsigned call = GET_CALL();
        const __amdgpu_buffer_rsrc_t srdP = __builtin_amdgcn_make_buffer_rsrc((void*)p.ws[dst], (short)0, (int)WS_CALL, 0x00020000);
        const unsigned dA = (call & 1u) * WS_PAR + rank * WS_SRC + wid * 65536u + (rg & 31u) * 2048u, dB = dA + 4 * 65536u;
        const unsigned cA = WS_CNT + (rank * 2u + ((rg >> 4) & 1u)) * 128u + wid * 4u, cB = cA + 16u;
        const unsigned pl = ldsb + P_BASE + wid * 2048u;
        unsigned a0 = ldsb + A_BASE + 128u * fr + 16u * (fq ^ (fr >> 1));
        unsigned a1 = ldsb + A_BASE + 128u * fr + 16u * ((fq + 4) ^ (fr >> 1));
        const unsigned q = (lane >> 2) & 3, pp = lane & 3;
        const unsigned fB = (q & 2) | ((fq & 1) << 2);
        unsigned bb[4], bh[4];
#pragma unroll
        for (int c = 0; c < 4; ++c) {
            bb[c] = ldsb + wid * 8192u + 128u * (8 * fq + q) + 16u * ((2 * c + (pp >> 1)) ^ fB) + 8u * (pp & 1);
            bh[c] = bb[c] + 2 * B_SLOT;
        }
        f32x4 acc[8][4];
        bf16x8 X0[8], X1[8];

        const int e0 = (int)wid * PUSH_GAP - 1, e1 = wid == 0 ? PUSH_D0 : e0 + PUSH_D;
        const int e2 = e0 + 4 * PUSH_GAP, e3 = e2 + PUSH_D;
        static_assert(PUSH_GAP >= 3 && PUSH_D < 4 * PUSH_GAP && PUSH_D0 >= 1 && PUSH_D0 < 4 * PUSH_GAP - 1 && 7 * PUSH_GAP + PUSH_D < 50, "push schedule");
        unsigned long long evm = 0;
        if (b < PUSH_WGS) {
            evm = (1ull << e1) | (1ull << e2) | (1ull << e3);
            if (wid == 0) {
                const char* const sp = (const char*)p.x + (size_t)(dst * 512u + (rg & 31u) * 16u) * 1024u;
                const unsigned vs0 = (lane >> 3) * 1024u + (((lane & 7) ^ (lane >> 4)) * 16u), vs1 = (vs0 ^ 64u) + 8192u;
                u32x4 d0, d1;
                asm volatile("global_load_dwordx4 %0, %2, %4 nt\n\tglobal_load_dwordx4 %1, %3, %4 nt\n\ts_waitcnt vmcnt(0)\n\t"
                             "buffer_store_dwordx4 %0, %5, %6, %7 offen sc0 sc1\n\tbuffer_store_dwordx4 %1, %5, %6, %7 offen offset:1024 sc0 sc1\n\ts_nop 1"
                             : "=&v"(d0), "=&v"(d1) : "v"(vs0), "v"(vs1), "s"(sp), "v"(vd), "s"(srdP), "s"(dA) : "memory");
            } else evm |= 1ull << e0;
        }
#define PUSH_EVENT(k) do { u32x4 d0, d1; unsigned t0, t1; asm volatile( \
            "s_cmp_eq_u32 %[kk], %[e1]\n\t" \
            "s_cbranch_scc1 1f\n\t" \
            "s_cmp_eq_u32 %[kk], %[e3]\n\t" \
            "s_cbranch_scc1 3f\n\t" \
            "v_add_u32 %[t0], %[pl], %[vd]\n\t" \
            "s_cmp_eq_u32 %[kk], %[e2]\n\t" \
            "s_cbranch_scc1 2f\n\t" \
            "ds_read_b128 %[d0], %[t0]\n\t" \
            "ds_read_b128 %[d1], %[t0] offset:1024\n\t" \
            "s_waitcnt lgkmcnt(0)\n\t" \
            "buffer_store_dwordx4 %[d0], %[vd], %[srd], %[dA] offen sc0 sc1\n\t" \
            "buffer_store_dwordx4 %[d1], %[vd], %[srd], %[dA] offen offset:1024 sc0 sc1\n\t" \
            "s_branch 8f\n" \
            "2:\n\t" \
            "ds_read_b128 %[d0], %[t0] offset:8192\n\t" \
            "ds_read_b128 %[d1], %[t0] offset:9216\n\t" \
            "s_waitcnt lgkmcnt(0)\n\t" \
            "buffer_store_dwordx4 %[d0], %[vd], %[srd], %[dB] offen sc0 sc1\n\t" \
            "buffer_store_dwordx4 %[d1], %[vd], %[srd], %[dB] offen offset:1024 sc0 sc1\n\t" \
            "s_branch 8f\n" \
            "1:\n\t" \
            "v_mov_b32 %[t0], 0\n\t" \
            "v_mov_b32 %[t1], 1\n\t" \
            "s_waitcnt vmcnt(0)\n\t" \
            "s_mov_b64 exec, 1\n\t" \
            "buffer_atomic_add %[t1], %[t0], %[srd], %[cA] offen sc1\n\t" \
            "s_mov_b64 exec, -1\n\t" \
            "s_branch 8f\n" \
            "3:\n\t" \
            "v_mov_b32 %[t0], 0\n\t" \
            "v_mov_b32 %[t1], 1\n\t" \
            "s_waitcnt vmcnt(0)\n\t" \
            "s_mov_b64 exec, 1\n\t" \
            "buffer_atomic_add %[t1], %[t0], %[srd], %[cB] offen sc1\n\t" \
            "s_mov_b64 exec, -1\n" \
            "8:\n\t" \
            "s_nop 1" \
            : [d0] "=&v"(d0), [d1] "=&v"(d1), [t0] "=&v"(t0), [t1] "=&v"(t1) \
            : [kk] "s"(k), [e1] "s"(e1), [e2] "s"(e2), [e3] "s"(e3), [vd] "v"(vd), [srd] "s"(srdP), [pl] "s"(pl), \
              [dA] "s"(dA), [dB] "s"(dB), [cA] "s"(cA), [cB] "s"(cB) : "memory", "scc"); } while (0)

        s16x4 Wl0[4], Wh0[4], Wl1[4], Wh1[4];
#define WFRAG(S, c) __builtin_bit_cast(bf16x8, __builtin_shufflevector(Wl##S[c], Wh##S[c], 0, 1, 2, 3, 4, 5, 6, 7))
#define MF(S, i) do { const int m_ = (i) >> 2, c_ = (m_ & 1) ? 3 - ((i) & 3) : ((i) & 3); \
            asm volatile("v_mfma_f32_16x16x32_bf16 %0, %1, %2, %0" : "+v"(acc[m_][c_]) : "v"(WFRAG(S, c_)), "v"(X##S[m_]) : "memory"); } while (0)
#define MF0(S, i) do { const int m_ = (i) >> 2, c_ = (m_ & 1) ? 3 - ((i) & 3) : ((i) & 3); \
            asm volatile("v_mfma_f32_16x16x32_bf16 %0, %1, %2, 0" : "=&v"(acc[m_][c_]) : "v"(WFRAG(S, c_)), "v"(X##S[m_]) : "memory"); } while (0)
#define RD(S, r, j) do { if ((j) < 8) { const unsigned o_ = ((r) == 2 ? bh[((j) >> 1) & 3] : bb[((j) >> 1) & 3] + (r) * B_SLOT) + (S) * 4096u + ((j) & 1) * 512u; \
                if ((j) & 1) Wh##S[((j) >> 1) & 3] = __builtin_amdgcn_ds_read_tr16_b64_v4i16((LDS s16x4*)o_); \
                else         Wl##S[((j) >> 1) & 3] = __builtin_amdgcn_ds_read_tr16_b64_v4i16((LDS s16x4*)o_); } \
            else X##S[(j) & 7] = *(const LDS bf16x8*)(((S) ? a1 : a0) + (r) * A_SLOT + 2048u * ((j) & 7)); } while (0)
#define RUN28(MFX, SM, i0, SR, r) do { _Pragma("unroll") for (int g = 0; g < 4; ++g) { \
            MFX(SM, (i0) + 7 * g); RD(SR, r, 4 * g); MFX(SM, (i0) + 7 * g + 1); MFX(SM, (i0) + 7 * g + 2); RD(SR, r, 4 * g + 1); \
            MFX(SM, (i0) + 7 * g + 3); MFX(SM, (i0) + 7 * g + 4); RD(SR, r, 4 * g + 2); MFX(SM, (i0) + 7 * g + 5); MFX(SM, (i0) + 7 * g + 6); RD(SR, r, 4 * g + 3); } } while (0)
#define RUN4(MFX, SM, i0) do { MFX(SM, i0); MFX(SM, (i0) + 1); MFX(SM, (i0) + 2); MFX(SM, (i0) + 3); } while (0)
#define TO_BARRIER() do { __builtin_amdgcn_s_waitcnt(0xC07F); ST_BEG(); BAR(); ST_END(st_bar); } while (0)
#define BODY(r, EV) do { RUN28(MF, 1, 4, 0, r); EV; RUN28(MF, 0, 0, 1, r); RUN4(MF, 0, 28); RUN4(MF, 1, 0); TO_BARRIER(); } while (0)
#define EVENT(mask, bit, k) do { if (__builtin_expect(((mask) >> (bit)) & 1u, 0)) PUSH_EVENT(k); } while (0)

        asm volatile("" : "+v"(a0), "+v"(a1), "+v"(bb[0]), "+v"(bb[1]), "+v"(bb[2]), "+v"(bb[3]), "+v"(bh[0]), "+v"(bh[1]), "+v"(bh[2]), "+v"(bh[3]));
        BAR();
        ST_FIRST();
#pragma unroll
        for (int j = 0; j < 16; ++j) RD(0, 0, j);
        RUN28(MF0, 0, 0, 1, 0); RUN4(MF0, 0, 28); RUN4(MF, 1, 0);
        TO_BARRIER();
        for (int kb = 1; kb < NSTEP; kb += 3) {
            unsigned em = (unsigned)(evm >> kb);
            asm volatile("" : "+s"(em));
            BODY(1, EVENT(em, 0, kb)); BODY(2, EVENT(em, 1, kb + 1)); BODY(0, EVENT(em, 2, kb + 2));
        }
        const unsigned tv = __hip_atomic_load(tmo, __ATOMIC_RELAXED, __HIP_MEMORY_SCOPE_AGENT);
#pragma unroll
        for (int i = 4; i < 32; ++i) MF(1, i);
        asm volatile("s_nop 7\n\ts_nop 1" ::: "memory");
        SB0();
        const bool bad = tv == call;
        char* orow = (char*)p.out + (size_t)(mq * 128 + fr) * WROW + colb + wid * 128u + (16 * (fq & 1) + 8 * (fq >> 1)) * 2;
#pragma unroll
        for (int m = 0; m < 8; ++m) {
            u32x4 o[2];
#pragma unroll
            for (int h = 0; h < 2; ++h) {
                const f32x4 u = acc[m][2 * h], v = acc[m][2 * h + 1];
                const unsigned u0 = pack_relu(u[0], u[1]), u1 = pack_relu(u[2], u[3]);
                const unsigned v0 = pack_relu(v[0], v[1]), v1 = pack_relu(v[2], v[3]);
                const auto s0 = __builtin_amdgcn_permlane16_swap(u0, v0, false, false);
                const auto s1 = __builtin_amdgcn_permlane16_swap(u1, v1, false, false);
                o[h] = (u32x4){s0[0], s1[0], s0[1], s1[1]};
                if (bad) o[h] = (u32x4){0x7fc07fc0u, 0x7fc07fc0u, 0x7fc07fc0u, 0x7fc07fc0u};
            }
            asm volatile("global_store_dwordx4 %0, %1, off sc0 sc1\n\tglobal_store_dwordx4 %0, %2, off offset:64 sc0 sc1\n\ts_nop 1"
                         :: "v"(orow + (size_t)m * 16 * WROW), "v"(o[0]), "v"(o[1]) : "memory");
        }
        BAR();
    }
}

}

extern "C" void kernel_launch(int rank, int nranks, ncclComm_t comm,
                              hipStream_t stream,
                              void* const* d_in, const int* in_sizes, int n_in,
                              void* d_out, int out_size,
                              const int* in_dtypes, int out_dtype,
                              void* const* d_out_all,
                              void* const* d_ws_all, size_t ws_bytes) {
    using namespace a2ag;
    if (nranks != NR || (unsigned)rank >= (unsigned)NR || n_in < 2 || ws_bytes < WS_END ||
        in_sizes[0] != 4096 * KSH || in_sizes[1] != KTOT * NTOT || out_size != MLOC * NTOT) {
        fprintf(stderr, "kernel_launch: unexpected shapes\n");
        return;
    }
    HIP_CHECK(hipFuncSetAttribute((const void*)a2a_gemm, hipFuncAttributeMaxDynamicSharedMemorySize, (int)LDS_BYTES));
    Params p;
    p.x = d_in[0]; p.w = d_in[1]; p.out = d_out; p.myws = (char*)d_ws_all[rank];
    for (int i = 0; i < NR; ++i) p.ws[i] = (char*)d_ws_all[i];
    p.rank = rank; p.pad = 0;
    hipLaunchKernelGGL(a2a_gemm, dim3(NWG), dim3(512), LDS_BYTES, stream, p);
    HIP_CHECK(hipGetLastError());
}
```

```cpp
#include <hip/hip_runtime.h>
#include <rccl/rccl.h>
#include <cstdio>

#define HIP_CHECK(expr) do { hipError_t _e = (expr); if (_e != hipSuccess) { \
    fprintf(stderr, "HIP error %s at %s:%d\n", hipGetErrorString(_e), __FILE__, __LINE__); \
    return; } } while (0)

namespace a2ag {

typedef __bf16 bf16x8 __attribute__((ext_vector_type(8)));
typedef __bf16 bf16x2 __attribute__((ext_vector_type(2)));
typedef short s16x4 __attribute__((ext_vector_type(4)));
typedef float f32x2 __attribute__((ext_vector_type(2)));
typedef float f32x4 __attribute__((ext_vector_type(4)));
typedef unsigned u32x4 __attribute__((ext_vector_type(4)));
#define LDS __attribute__((address_space(3)))
#define DI __device__ __forceinline__

constexpr int NR = 8;
constexpr int MLOC = 512;
constexpr int KSH = 512;
constexpr int KTOT = 4096, NTOT = 16384;
constexpr int NSTEP = KTOT / 64;
constexpr int NWG = 256;
constexpr unsigned B_SLOT = 32768, A_SLOT = 16384;
constexpr unsigned A_BASE = 3 * B_SLOT;
constexpr unsigned LDS_BYTES = A_BASE + 3 * A_SLOT;
constexpr unsigned X_BYTES = 4096u * KSH * 2u;
constexpr unsigned W_BYTES = (unsigned)KTOT * NTOT * 2u;
constexpr unsigned WROW = NTOT * 2u;

constexpr unsigned WS_SRC = 512u * 1024u;
constexpr unsigned WS_PAR = NR * WS_SRC;
constexpr unsigned WS_CNT = 2 * WS_PAR;
constexpr unsigned WS_CALL = WS_CNT + NR * 2 * 128;
constexpr unsigned WS_TMO = WS_CALL + NWG * 128;
constexpr unsigned WS_END = WS_TMO + 128;
constexpr unsigned PUSH_WGS = 7 * 32;
constexpr unsigned CNT_PER_CALL = 16;
constexpr unsigned long long DEADLINE_TICKS = 50000000ull;

#define PUSH_GAP 2
#define PUSH_STAG 0
#define PUSH_DMIN 2
#define PUSH_DMAX 6
#define KORD 2

#define ST_DECL
#define ST_BEG()
#define ST_END(acc)
#define ST_FIRST()
#define ST_OUT(x6)

struct Params {
    const void* x; const void* w; void* out; char* myws;
    char* ws[NR];
    int rank; int pad;
};

#define BAR() do { asm volatile("" ::: "memory"); __builtin_amdgcn_s_barrier(); asm volatile("" ::: "memory"); } while (0)
#define WAITV(n) asm volatile("s_waitcnt vmcnt(" #n ")" ::: "memory")
#define SB0() __builtin_amdgcn_sched_barrier(0)

DI unsigned rfl(unsigned v) { return __builtin_amdgcn_readfirstlane(v); }

#define DMA4(suffix, lds, v, srd, soff) asm volatile("s_mov_b32 m0, %0\n\ts_nop 0\n\t" \
        "buffer_load_dwordx4 %1, %5, %6 offen " suffix " lds\n\t" \
        "buffer_load_dwordx4 %2, %5, %6 offen offset:1024 " suffix " lds\n\t" \
        "buffer_load_dwordx4 %3, %5, %6 offen offset:2048 " suffix " lds\n\t" \
        "buffer_load_dwordx4 %4, %5, %6 offen offset:3072 " suffix " lds" \
        :: "s"(lds), "v"((v)[0]), "v"((v)[1]), "v"((v)[2]), "v"((v)[3]), "s"(srd), "s"(soff) : "memory")

DI void step_map(unsigned t, unsigned& j, unsigned& kc) {
    if (t < 8) { j = 0; kc = t; }
    else { const unsigned u = t - 8; kc = (u * 147u) >> 10; j = 1 + (u - 7 * kc); }
}

DI unsigned stage_map(unsigned s, unsigned q) {
    const unsigned o = (q + 2u) & 3u, v = (s - o) & 3u, s0 = s - v;
    if (s < o || s0 + 3u >= NSTEP || (s0 < 8u && s0 + 2u >= 8u)) return s;
    return v == 0 ? s + 2u : (v == 3 ? s : s - 1u);
}

DI unsigned pack_relu(float a, float b) {
    f32x2 v = {__builtin_elementwise_maximum(a, 0.f), __builtin_elementwise_maximum(b, 0.f)};
    bf16x2 r = __builtin_convertvector(v, bf16x2);
    return __builtin_bit_cast(unsigned, r);
}

__global__ __launch_bounds__(512, 2) __attribute__((amdgpu_num_vgpr(232))) void a2a_gemm(Params p) {
    extern __shared__ __attribute__((aligned(16))) char smem[];
    const unsigned ldsb = (unsigned)(size_t)(LDS char*)smem;
    const unsigned tid = threadIdx.x, lane = tid & 63;
    const unsigned wid = rfl(tid >> 6);
    const unsigned b = blockIdx.x;
    const unsigned mq = (b >> 3) & 3, mh = mq >> 1;
    const unsigned colb = (b & 7) * 4096u + (b >> 5) * 512u;
    const unsigned rank = (unsigned)p.rank & (NR - 1);
    const unsigned rg = b / 7u, dst = (rank + 1u + (b - rg * 7u)) & (NR - 1);
    char* const myws = p.myws;
    unsigned* const callp = (unsigned*)(myws + WS_CALL + b * 128);
    unsigned* const tmo = (unsigned*)(myws + WS_TMO);
#define GET_CALL() (rfl(*(const volatile unsigned __attribute__((address_space(4)))*)(size_t)callp) + 1u)
    ST_DECL;

    if (wid >= 4) {
        const unsigned lw = wid - 4;
        const __amdgpu_buffer_rsrc_t srdW = __builtin_amdgcn_make_buffer_rsrc((void*)p.w, (short)0, (int)W_BYTES, 0x00020000);
        const __amdgpu_buffer_rsrc_t srdX = __builtin_amdgcn_make_buffer_rsrc((void*)p.x, (short)0, (int)X_BYTES, 0x00020000);
        const __amdgpu_buffer_rsrc_t srdR = __builtin_amdgcn_make_buffer_rsrc((void*)myws, (short)0, (int)WS_CNT, 0x00020000);
        unsigned vW[4], vL[4];
        const unsigned vR = lane * 16u;
#pragma unroll
        for (int j = 0; j < 4; ++j) {
            vW[j] = (lane >> 3) * WROW + (((lane & 7) ^ (((lane >> 3) & 2) | ((j & 1) << 2))) * 16u) + j * (8u * WROW - 1024u);
            vL[j] = (lane >> 3) * 1024u + (((lane & 7) ^ (4 * (j & 1) + (lane >> 4))) * 16u) + j * (8192u - 1024u);
        }
        const unsigned sW = colb + lw * 128u;
        const unsigned lW = ldsb + lw * 8192u;
        const unsigned sL = (rank * 512u + mq * 128u + lw * 32u) * 1024u;
        const unsigned lA = ldsb + A_BASE + lw * 4096u;
        unsigned tW, tA;
        { unsigned j, kc; step_map(stage_map(lane, mq), j, kc);
          const unsigned src = (rank + j) & (NR - 1);
          tW = sW + (src * KSH + kc * 64u) * WROW;
          tA = j == 0 ? sL + 128u * kc : mq * 16384u + lw * 4096u + src * WS_SRC + 65536u * kc; }
#define RL(t, s) __builtin_amdgcn_readlane(t, s)
#define ISSUE_W(mw, ow) do { DMA4("", mw, vW, srdW, ow); DMA4("", (mw) + 4096u, vW, srdW, (ow) + 32u * WROW); } while (0)
        __amdgpu_buffer_rsrc_t srdA = srdX;
        unsigned vA[4] = {vL[0], vL[1], vL[2], vL[3]};
#define ISSUE(mw, ow, ma, oa) do { ISSUE_W(mw, ow); DMA4("sc0 sc1", ma, vA, srdA, oa); } while (0)
        { const unsigned w0 = RL(tW, 0), a0 = RL(tA, 0), w1 = RL(tW, 1), a1 = RL(tA, 1);
          asm volatile("s_nop 4" ::: "memory");
          ISSUE(lW, w0, lA, a0); ISSUE(lW + B_SLOT, w1, lA + A_SLOT, a1); }
        unsigned ow = RL(tW, 2), oa = RL(tA, 2);
        const unsigned call = GET_CALL();
        WAITV(12);
        BAR();
        ISSUE(lW + 2 * B_SLOT, ow, lA + 2 * A_SLOT, oa);
        ow = RL(tW, 3); oa = RL(tA, 3);
        if (lw == 3) {
            const unsigned vp = (rank * 512u + mq * 128u + (b >> 5) * 16u + (lane & 15)) * 1024u + (4u + (lane >> 4)) * 128u;
            asm volatile("buffer_load_dword v220, %0, %1, 0 offen" :: "v"(vp), "s"(srdX) : "memory", "v220");
        }
        if (lane >= 8) tA += (call & 1u) * WS_PAR;
        const unsigned target = call * CNT_PER_CALL;
        unsigned long long ready = 0xffull;
        const unsigned* cptr;
        { unsigned j, kc; step_map(stage_map(8u + (lane < 56 ? lane : 55u), mq), j, kc);
          cptr = (const unsigned*)(myws + WS_CNT + (((rank + j) & (NR - 1)) * 2u + mh) * 128u + kc * 4u); }
        unsigned ksamp = 3;
        unsigned long long ev = (1ull << 61) | (1ull << 5);
        if (lw == 0 && ~ready != 0) ev |= 3ull << 3;
        unsigned mw = lW, ma = lA, k = 0;
#define ST_LINE()
#define STEP() do { ISSUE(mw, ow, ma, oa); ow = RL(tW, k + 4); oa = RL(tA, k + 4); ST_LINE(); \
            mw = (mw == lW + 2 * B_SLOT) ? lW : mw + B_SLOT; ma = (ma == lA + 2 * A_SLOT) ? lA : ma + A_SLOT; ++k; \
            asm volatile("" : "+s"(ow), "+s"(oa), "+s"(mw), "+s"(ma), "+s"(k)); } while (0)
        asm volatile("" : "+s"(ow), "+s"(oa), "+s"(mw), "+s"(ma), "+s"(k));
        WAITV(12);
        BAR();
        for (;;) {
            const unsigned kend = rfl(k + (unsigned)__builtin_ctzll(ev >> k));
            while (k != kend) { STEP(); WAITV(12); BAR(); }
            if (k == NSTEP - 3) break;
            ev &= ~(1ull << k);
            if (k == 5) { srdA = srdR; vA[0] = vA[1] = vA[2] = vA[3] = vR; }
            if (lw == 0 && k == ksamp) {
                asm volatile("global_load_dword v221, %0, off sc0 sc1" :: "v"(cptr) : "memory", "v221");
                STEP(); WAITV(13);
            } else {
                STEP(); WAITV(12);
                if (lw == 0 && k == ksamp + 2) {
                    unsigned pv; asm volatile("v_mov_b32 %0, v221" : "=v"(pv) :: "memory");
                    ready |= __ballot(pv >= target) << 8;
                    if (!((ready >> (k + 3)) & 1ull)) {
                        const unsigned long long t0 = __builtin_amdgcn_s_memrealtime();
                        for (;;) {
                            unsigned v, to;
                            asm volatile("global_load_dword %0, %2, off sc0 sc1\n\tglobal_load_dword %1, %3, off sc1\n\ts_waitcnt vmcnt(0)"
                                         : "=&v"(v), "=&v"(to) : "v"(cptr), "v"(tmo) : "memory");
                            ready |= __ballot(v >= target) << 8;
                            if ((ready >> (k + 3)) & 1ull) break;
                            if (rfl(to) == call || __builtin_amdgcn_s_memrealtime() - t0 > DEADLINE_TICKS) {
                                asm volatile("global_store_dword %0, %1, off sc1\n\ts_waitcnt vmcnt(0)" :: "v"(tmo), "v"(call) : "memory");
                                ready = ~0ull;
                                break;
                            }
                            __builtin_amdgcn_s_sleep(2);
                        }
                    }
                    if (~ready != 0) {
                        const unsigned S = (unsigned)__builtin_ctzll(~ready);
                        const unsigned km = (S - 5u) - ((S - 5u + 1u - mq) & 3u);
                        ksamp = (int)km >= (int)k ? km : S - 5u;
                        ev |= ((ksamp >= k ? 1ull : 0ull) << ksamp) | (2ull << ksamp);
                    }
                }
            }
            BAR();
        }
        WAITV(0);
        BAR(); BAR(); BAR();
        if (lw == 0 && lane == 0) __hip_atomic_store(callp, call, __ATOMIC_RELAXED, __HIP_MEMORY_SCOPE_AGENT);
        ST_OUT(st_pollk);
    } else {
        const unsigned fr = lane & 15, fq = lane >> 4;
        const unsigned call = GET_CALL();
        unsigned evm = 0;
        char* dp = nullptr; char* cp = nullptr;
        const int e2 = PUSH_GAP * ((int)wid + 4) - 1 + (PUSH_STAG ? (int)(rg % PUSH_GAP) : 0), e0 = wid ? e2 - 4 * PUSH_GAP : -1;
        static_assert(PUSH_GAP >= 2 && 8 * PUSH_GAP + PUSH_DMAX < 32 && (1 + PUSH_STAG) * PUSH_GAP + PUSH_DMAX <= 10 && PUSH_DMAX < 4 * PUSH_GAP, "push schedule");
        if (b < PUSH_WGS) {
            char* const pws = p.ws[dst];
            const char* const sp = (const char*)p.x + (size_t)(dst * 512u + (rg & 31u) * 16u) * 1024u + wid * 128u;
            const unsigned vs0 = (lane >> 3) * 1024u + (((lane & 7) ^ (lane >> 4)) * 16u), vs1 = (vs0 ^ 64u) + 8192u;
            cp = pws + WS_CNT + (rank * 2u + ((rg >> 4) & 1u)) * 128u + wid * 4u;
            dp = pws + (call & 1u) * WS_PAR + rank * WS_SRC + wid * 65536u + (rg & 31u) * 2048u;
            asm volatile("v_mov_b32 v248, %0\n\tv_mov_b32 v249, 0\n\tv_mov_b32 v250, 1" :: "v"(lane * 16u) : "v248", "v249", "v250");
            if (wid == 0) {
                asm volatile("global_load_dwordx4 v[232:235], %0, %2 nt\n\tglobal_load_dwordx4 v[236:239], %1, %2 nt\n\t"
                             "s_mov_b64 exec, 1\n\tglobal_atomic_add v249, v249, %3 sc1\n\tglobal_atomic_add v249, v249, %4 offset:64 sc1\n\ts_mov_b64 exec, -1\n\t"
                             "s_waitcnt vmcnt(2)\n\tglobal_store_dwordx4 v248, v[232:235], %3 sc0 sc1\n\tglobal_store_dwordx4 v248, v[236:239], %3 offset:1024 sc0 sc1"
                             :: "v"(vs0), "v"(vs1), "s"(sp), "s"(dp), "s"(cp) : "memory", "v232", "v233", "v234", "v235", "v236", "v237", "v238", "v239");
                evm = 0x1eu;
            } else {
                asm volatile("global_load_dwordx4 v[232:235], %0, %2 nt\n\tglobal_load_dwordx4 v[236:239], %1, %2 nt"
                             :: "v"(vs0), "v"(vs1), "s"(sp) : "memory", "v232", "v233", "v234", "v235", "v236", "v237", "v238", "v239");
                evm = (1u << e0) | (((2u << (PUSH_DMAX - PUSH_DMIN)) - 1u) << (e0 + PUSH_DMIN));
            }
            asm volatile("global_load_dwordx4 v[240:243], %0, %2 offset:512 nt\n\tglobal_load_dwordx4 v[244:247], %1, %2 offset:512 nt"
                         :: "v"(vs0), "v"(vs1), "s"(sp) : "memory", "v240", "v241", "v242", "v243", "v244", "v245", "v246", "v247");
            evm |= (1u << e2) | (((2u << (PUSH_DMAX - PUSH_DMIN)) - 1u) << (e2 + PUSH_DMIN));
        }
        auto push_event = [&](int k) {
            if (!((evm >> k) & 1u)) return;
            if (k == e0) asm volatile("s_waitcnt vmcnt(0)\n\tglobal_store_dwordx4 v248, v[232:235], %0 sc0 sc1\n\tglobal_store_dwordx4 v248, v[236:239], %0 offset:1024 sc0 sc1" :: "s"(dp) : "memory");
            else if (k == e2) asm volatile("s_waitcnt vmcnt(0)\n\tglobal_store_dwordx4 v248, v[240:243], %0 sc0 sc1\n\tglobal_store_dwordx4 v248, v[244:247], %0 offset:1024 sc0 sc1" :: "s"(dp + 4 * 65536u) : "memory");
            else {
                const unsigned pend = __builtin_amdgcn_s_getreg((3 << 11) | 7);
                if (pend == 0 || !((evm >> (k + 1)) & 1u) || k + 1 == e2) {
                    asm volatile("s_waitcnt vmcnt(0)\n\ts_mov_b64 exec, 1\n\tglobal_atomic_add v249, v250, %0 sc1\n\ts_mov_b64 exec, -1" :: "s"(k > e2 ? cp + 16 : cp) : "memory");
                    evm &= k > e2 ? 0u : ~0u << e2;
                }
            }
        };
        unsigned a0 = ldsb + A_BASE + 128u * fr + 16u * (fq ^ (fr >> 1));
        unsigned a1 = ldsb + A_BASE + 128u * fr + 16u * ((fq + 4) ^ (fr >> 1));
        const unsigned q = (lane >> 2) & 3, pp = lane & 3;
        const unsigned fB = (q & 2) | ((fq & 1) << 2);
        unsigned bb[4], bh[4];
#pragma unroll
        for (int c = 0; c < 4; ++c) {
            bb[c] = ldsb + wid * 8192u + 128u * (8 * fq + q) + 16u * ((2 * c + (pp >> 1)) ^ fB) + 8u * (pp & 1);
            bh[c] = bb[c] + 2 * B_SLOT;
        }
        f32x4 acc[8][4];
        bf16x8 X0[8], X1[8];

        s16x4 Wl0[4], Wh0[4], Wl1[4], Wh1[4];
#define WFRAG(S, c) __builtin_bit_cast(bf16x8, __builtin_shufflevector(Wl##S[c], Wh##S[c], 0, 1, 2, 3, 4, 5, 6, 7))
#define MF(S, i) do { const int m_ = (i) >> 2, c_ = (m_ & 1) ? 3 - ((i) & 3) : ((i) & 3); \
            asm volatile("v_mfma_f32_16x16x32_bf16 %0, %1, %2, %0" : "+v"(acc[m_][c_]) : "v"(WFRAG(S, c_)), "v"(X##S[m_]) : "memory"); } while (0)
#define MF0(S, i) do { const int m_ = (i) >> 2, c_ = (m_ & 1) ? 3 - ((i) & 3) : ((i) & 3); \
            asm volatile("v_mfma_f32_16x16x32_bf16 %0, %1, %2, 0" : "=&v"(acc[m_][c_]) : "v"(WFRAG(S, c_)), "v"(X##S[m_]) : "memory"); } while (0)
#define RD(S, r, j) do { if ((j) < 8) { const unsigned o_ = ((r) == 2 ? bh[((j) >> 1) & 3] : bb[((j) >> 1) & 3] + (r) * B_SLOT) + (S) * 4096u + ((j) & 1) * 512u; \
                if ((j) & 1) Wh##S[((j) >> 1) & 3] = __builtin_amdgcn_ds_read_tr16_b64_v4i16((LDS s16x4*)o_); \
                else         Wl##S[((j) >> 1) & 3] = __builtin_amdgcn_ds_read_tr16_b64_v4i16((LDS s16x4*)o_); } \
            else X##S[(j) & 7] = *(const LDS bf16x8*)(((S) ? a1 : a0) + (r) * A_SLOT + 2048u * ((j) & 7)); } while (0)
#define RUN28(MFX, SM, i0, SR, r) do { _Pragma("unroll") for (int g = 0; g < 4; ++g) { \
            MFX(SM, (i0) + 7 * g); RD(SR, r, 4 * g); MFX(SM, (i0) + 7 * g + 1); MFX(SM, (i0) + 7 * g + 2); RD(SR, r, 4 * g + 1); \
            MFX(SM, (i0) + 7 * g + 3); MFX(SM, (i0) + 7 * g + 4); RD(SR, r, 4 * g + 2); MFX(SM, (i0) + 7 * g + 5); MFX(SM, (i0) + 7 * g + 6); RD(SR, r, 4 * g + 3); } } while (0)
#define RUN4(MFX, SM, i0) do { MFX(SM, i0); MFX(SM, (i0) + 1); MFX(SM, (i0) + 2); MFX(SM, (i0) + 3); } while (0)
#define TO_BARRIER() do { __builtin_amdgcn_s_waitcnt(0xC07F); ST_BEG(); BAR(); ST_END(st_bar); } while (0)
#define EVPOS 2
#define BODY(r, i) do { RUN28(MF, 1, 4, 0, r); RUN28(MF, 0, 0, 1, r); RUN4(MF, 0, 28); \
            _Pragma("unroll") for (int j = 0; j < EVPOS; ++j) MF(1, j); \
            if (__builtin_expect((em >> (i)) & 1u, 0)) push_event(kb + (i)); \
            _Pragma("unroll") for (int j = EVPOS; j < 4; ++j) MF(1, j); TO_BARRIER(); } while (0)

        asm volatile("" : "+v"(a0), "+v"(a1), "+v"(bb[0]), "+v"(bb[1]), "+v"(bb[2]), "+v"(bb[3]), "+v"(bh[0]), "+v"(bh[1]), "+v"(bh[2]), "+v"(bh[3]));
        BAR();
        ST_FIRST();
#pragma unroll
        for (int j = 0; j < 16; ++j) RD(0, 0, j);
        RUN28(MF0, 0, 0, 1, 0); RUN4(MF0, 0, 28); RUN4(MF, 1, 0);
        TO_BARRIER();
        for (int kb = 1; kb < NSTEP; kb += 3) {
            unsigned em = evm >> (kb & 31);
            asm volatile("" : "+s"(em));
            BODY(1, 0); BODY(2, 1); BODY(0, 2);
        }
        const unsigned tv = __hip_atomic_load(tmo, __ATOMIC_RELAXED, __HIP_MEMORY_SCOPE_AGENT);
#pragma unroll
        for (int i = 4; i < 32; ++i) MF(1, i);
        asm volatile("s_nop 7\n\ts_nop 1" ::: "memory");
        SB0();
        const bool bad = tv == call;
        char* orow = (char*)p.out + (size_t)(mq * 128 + fr) * WROW + colb + wid * 128u + (16 * (fq & 1) + 8 * (fq >> 1)) * 2;
#pragma unroll
        for (int m = 0; m < 8; ++m) {
            u32x4 o[2];
#pragma unroll
            for (int h = 0; h < 2; ++h) {
                const f32x4 u = acc[m][2 * h], v = acc[m][2 * h + 1];
                const unsigned u0 = pack_relu(u[0], u[1]), u1 = pack_relu(u[2], u[3]);
                const unsigned v0 = pack_relu(v[0], v[1]), v1 = pack_relu(v[2], v[3]);
                const auto s0 = __builtin_amdgcn_permlane16_swap(u0, v0, false, false);
                const auto s1 = __builtin_amdgcn_permlane16_swap(u1, v1, false, false);
                o[h] = (u32x4){s0[0], s1[0], s0[1], s1[1]};
                if (bad) o[h] = (u32x4){0x7fc07fc0u, 0x7fc07fc0u, 0x7fc07fc0u, 0x7fc07fc0u};
            }
            asm volatile("global_store_dwordx4 %0, %1, off sc0 sc1\n\tglobal_store_dwordx4 %0, %2, off offset:64 sc0 sc1\n\ts_nop 1"
                         :: "v"(orow + (size_t)m * 16 * WROW), "v"(o[0]), "v"(o[1]) : "memory");
        }
        BAR();
    }
}

}

extern "C" void kernel_launch(int rank, int nranks, ncclComm_t comm,
                              hipStream_t stream,
                              void* const* d_in, const int* in_sizes, int n_in,
                              void* d_out, int out_size,
                              const int* in_dtypes, int out_dtype,
                              void* const* d_out_all,
                              void* const* d_ws_all, size_t ws_bytes) {
    using namespace a2ag;
    if (nranks != NR || (unsigned)rank >= (unsigned)NR || n_in < 2 || ws_bytes < WS_END ||
        in_sizes[0] != 4096 * KSH || in_sizes[1] != KTOT * NTOT || out_size != MLOC * NTOT) {
        fprintf(stderr, "kernel_launch: unexpected shapes\n");
        return;
    }
    HIP_CHECK(hipFuncSetAttribute((const void*)a2a_gemm, hipFuncAttributeMaxDynamicSharedMemorySize, (int)LDS_BYTES));
    Params p;
    p.x = d_in[0]; p.w = d_in[1]; p.out = d_out; p.myws = (char*)d_ws_all[rank];
    for (int i = 0; i < NR; ++i) p.ws[i] = (char*)d_ws_all[i];
    p.rank = rank; p.pad = 0;
    hipLaunchKernelGGL(a2a_gemm, dim3(NWG), dim3(512), LDS_BYTES, stream, p);
    HIP_CHECK(hipGetLastError());
}
```

```cpp
#include <hip/hip_runtime.h>
#include <rccl/rccl.h>
#include <cstdio>

#define HIP_CHECK(expr) do { hipError_t _e = (expr); if (_e != hipSuccess) { \
    fprintf(stderr, "HIP error %s at %s:%d\n", hipGetErrorString(_e), __FILE__, __LINE__); \
    return; } } while (0)

namespace a2ag {

typedef __bf16 bf16x8 __attribute__((ext_vector_type(8)));
typedef __bf16 bf16x2 __attribute__((ext_vector_type(2)));
typedef short s16x4 __attribute__((ext_vector_type(4)));
typedef float f32x2 __attribute__((ext_vector_type(2)));
typedef float f32x4 __attribute__((ext_vector_type(4)));
typedef unsigned u32x4 __attribute__((ext_vector_type(4)));
#define LDS __attribute__((address_space(3)))
#define DI __device__ __forceinline__

constexpr int NR = 8;
constexpr int MLOC = 512;
constexpr int KSH = 512;
constexpr int KTOT = 4096, NTOT = 16384;
constexpr int NSTEP = KTOT / 64;
constexpr int NWG = 256;
constexpr unsigned B_SLOT = 32768, A_SLOT = 16384;
constexpr unsigned A_BASE = 3 * B_SLOT;
constexpr unsigned LDS_BYTES = A_BASE + 3 * A_SLOT;
constexpr unsigned X_BYTES = 4096u * KSH * 2u;
constexpr unsigned W_BYTES = (unsigned)KTOT * NTOT * 2u;
constexpr unsigned WROW = NTOT * 2u;

constexpr unsigned WS_SRC = 512u * 1024u;
constexpr unsigned WS_PAR = NR * WS_SRC;
constexpr unsigned WS_CNT = 2 * WS_PAR;
constexpr unsigned WS_CALL = WS_CNT + NR * 4 * 128;
constexpr unsigned WS_TMO = WS_CALL + NWG * 128;
constexpr unsigned WS_END = WS_TMO + 128;
#define PUSH_E 4
constexpr unsigned PUSH_G = 8 / PUSH_E, PUSH_P = 4 / PUSH_E;
constexpr unsigned CNT_PER_CALL = 4 * PUSH_E;
constexpr unsigned long long DEADLINE_TICKS = 50000000ull;

#define PUSH_GAP 3
#define PUSH_D 5
#define KORD 2

#define ST_DECL
#define ST_BEG()
#define ST_END(acc)
#define ST_FIRST()
#define ST_OUT(x6)

struct Params {
    const void* x; const void* w; void* out; char* myws;
    char* ws[NR];
    int rank; int pad;
};

#define BAR() do { asm volatile("" ::: "memory"); __builtin_amdgcn_s_barrier(); asm volatile("" ::: "memory"); } while (0)
#define WAITV(n) asm volatile("s_waitcnt vmcnt(" #n ")" ::: "memory")
#define SB0() __builtin_amdgcn_sched_barrier(0)

DI unsigned rfl(unsigned v) { return __builtin_amdgcn_readfirstlane(v); }

#define DMA4(suffix, lds, v, srd, soff) asm volatile("s_mov_b32 m0, %0\n\ts_nop 0\n\t" \
        "buffer_load_dwordx4 %1, %5, %6 offen " suffix " lds\n\t" \
        "buffer_load_dwordx4 %2, %5, %6 offen offset:1024 " suffix " lds\n\t" \
        "buffer_load_dwordx4 %3, %5, %6 offen offset:2048 " suffix " lds\n\t" \
        "buffer_load_dwordx4 %4, %5, %6 offen offset:3072 " suffix " lds" \
        :: "s"(lds), "v"((v)[0]), "v"((v)[1]), "v"((v)[2]), "v"((v)[3]), "s"(srd), "s"(soff) : "memory")

DI void step_map(unsigned t, unsigned& j, unsigned& kc) {
    if (t < 8) { j = 0; kc = t; }
    else { const unsigned u = t - 8; kc = (u * 147u) >> 10; j = 1 + (u - 7 * kc); }
}

DI unsigned stage_map(unsigned s, unsigned q) {
    const unsigned o = (q + 2u) & 3u, v = (s - o) & 3u, s0 = s - v;
    if (s < o || s0 + 3u >= NSTEP || (s0 < 8u && s0 + 2u >= 8u)) return s;
    return v == 0 ? s + 2u : (v == 3 ? s : s - 1u);
}

DI unsigned pack_relu(float a, float b) {
    f32x2 v = {__builtin_elementwise_maximum(a, 0.f), __builtin_elementwise_maximum(b, 0.f)};
    bf16x2 r = __builtin_convertvector(v, bf16x2);
    return __builtin_bit_cast(unsigned, r);
}

__global__ __launch_bounds__(512, 2) __attribute__((amdgpu_num_vgpr(232))) void a2a_gemm(Params p) {
    extern __shared__ __attribute__((aligned(16))) char smem[];
    const unsigned ldsb = (unsigned)(size_t)(LDS char*)smem;
    const unsigned tid = threadIdx.x, lane = tid & 63;
    const unsigned wid = rfl(tid >> 6);
    const unsigned b = blockIdx.x;
    const unsigned mq = (b >> 3) & 3;
    const unsigned colb = (b & 7) * 4096u + (b >> 5) * 512u;
    const unsigned rank = (unsigned)p.rank & (NR - 1);
    const unsigned pi = (b >> 3) - (b >> 6), pq = (pi * 37u) >> 8;
    const bool pusher = ((b >> 3) & 7u) != 7u;
    const unsigned dst = (rank + 1u + (pi - pq * 7u)) & (NR - 1);
    char* const myws = p.myws;
    unsigned* const callp = (unsigned*)(myws + WS_CALL + b * 128);
    unsigned* const tmo = (unsigned*)(myws + WS_TMO);
#define GET_CALL() (rfl(*(const volatile unsigned __attribute__((address_space(4)))*)(size_t)callp) + 1u)
    ST_DECL;

    if (wid >= 4) {
        const unsigned lw = wid - 4;
        const __amdgpu_buffer_rsrc_t srdW = __builtin_amdgcn_make_buffer_rsrc((void*)p.w, (short)0, (int)W_BYTES, 0x00020000);
        const __amdgpu_buffer_rsrc_t srdX = __builtin_amdgcn_make_buffer_rsrc((void*)p.x, (short)0, (int)X_BYTES, 0x00020000);
        const __amdgpu_buffer_rsrc_t srdR = __builtin_amdgcn_make_buffer_rsrc((void*)myws, (short)0, (int)WS_CNT, 0x00020000);
#define WLAY 1
        unsigned vW[4], vW2[4], vL[4];
        const unsigned vR = lane * 16u;
#pragma unroll
        for (int j = 0; j < 4; ++j) {
            vW[j] = (lane >> 5) * WROW + ((((lane >> 3) & 3) ^ (lane >> 5)) * 128u) + (((lane & 7) ^ ((j & 1) << 1)) * 16u) + j * (2u * WROW - 1024u);
            vW2[j] = vW[j] ^ 64u;
            vL[j] = (lane >> 3) * 1024u + (((lane & 7) ^ (4 * (j & 1) + (lane >> 4))) * 16u) + j * (8192u - 1024u);
        }
        const unsigned sW = colb + lw * 16u * WROW;
        constexpr unsigned WQ2 = 8u * WROW;
        const unsigned lW = ldsb + lw * 8192u;
        const unsigned sL = (rank * 512u + mq * 128u + lw * 32u) * 1024u;
        const unsigned lA = ldsb + A_BASE + lw * 4096u;
        unsigned tW, tA;
        { unsigned j, kc; step_map(stage_map(lane, mq), j, kc);
          const unsigned src = (rank + j) & (NR - 1);
          tW = sW + (src * KSH + kc * 64u) * WROW;
          tA = j == 0 ? sL + 128u * kc : mq * 16384u + lw * 4096u + src * WS_SRC + 65536u * kc; }
#define RL(t, s) __builtin_amdgcn_readlane(t, s)
#define ISSUE_W(mw, ow) do { DMA4("", mw, vW, srdW, ow); DMA4("", (mw) + 4096u, vW2, srdW, (ow) + WQ2); } while (0)
        __amdgpu_buffer_rsrc_t srdA = srdX;
        unsigned vA[4] = {vL[0], vL[1], vL[2], vL[3]};
#define ISSUE(mw, ow, ma, oa) do { ISSUE_W(mw, ow); DMA4("sc0 sc1", ma, vA, srdA, oa); } while (0)
        { const unsigned w0 = RL(tW, 0), a0 = RL(tA, 0), w1 = RL(tW, 1), a1 = RL(tA, 1);
          asm volatile("s_nop 4" ::: "memory");
          ISSUE(lW, w0, lA, a0); ISSUE(lW + B_SLOT, w1, lA + A_SLOT, a1); }
        unsigned ow = RL(tW, 2), oa = RL(tA, 2);
        const unsigned call = GET_CALL();
        WAITV(12);
        BAR();
        ISSUE(lW + 2 * B_SLOT, ow, lA + 2 * A_SLOT, oa);
        ow = RL(tW, 3); oa = RL(tA, 3);
        if (lw == 3) {
            const unsigned vp = (rank * 512u + mq * 128u + (b >> 5) * 16u + (lane & 15)) * 1024u + (4u + (lane >> 4)) * 128u;
            asm volatile("buffer_load_dword v220, %0, %1, 0 offen" :: "v"(vp), "s"(srdX) : "memory", "v220");
        }
        if (lane >= 8) tA += (call & 1u) * WS_PAR;
        const unsigned target = call * CNT_PER_CALL;
        unsigned long long ready = 0xffull;
        const unsigned* cptr;
        { unsigned j, kc; step_map(stage_map(8u + (lane < 56 ? lane : 55u), mq), j, kc);
          cptr = (const unsigned*)(myws + WS_CNT + (((rank + j) & (NR - 1)) * 4u + mq) * 128u + kc * 4u); }
        unsigned ksamp = 3;
        unsigned long long ev = (1ull << 61) | (1ull << 5);
        if (lw == 0 && ~ready != 0) ev |= 3ull << 3;
        unsigned mw = lW, ma = lA, k = 0;
#define ST_LINE()
#define STEP() do { ISSUE(mw, ow, ma, oa); ow = RL(tW, k + 4); oa = RL(tA, k + 4); ST_LINE(); \
            mw = (mw == lW + 2 * B_SLOT) ? lW : mw + B_SLOT; ma = (ma == lA + 2 * A_SLOT) ? lA : ma + A_SLOT; ++k; \
            asm volatile("" : "+s"(ow), "+s"(oa), "+s"(mw), "+s"(ma), "+s"(k)); } while (0)
        asm volatile("" : "+s"(ow), "+s"(oa), "+s"(mw), "+s"(ma), "+s"(k));
        WAITV(12);
        BAR();
        for (;;) {
            const unsigned kend = rfl(k + (unsigned)__builtin_ctzll(ev >> k));
            while (k != kend) { STEP(); WAITV(12); BAR(); }
            if (k == NSTEP - 3) break;
            ev &= ~(1ull << k);
            if (k == 5) { srdA = srdR; vA[0] = vA[1] = vA[2] = vA[3] = vR; }
            if (lw == 0 && k == ksamp) {
                asm volatile("global_load_dword v221, %0, off sc0 sc1" :: "v"(cptr) : "memory", "v221");
                STEP(); WAITV(13);
            } else {
                STEP(); WAITV(12);
                if (lw == 0 && k == ksamp + 2) {
                    unsigned pv; asm volatile("v_mov_b32 %0, v221" : "=v"(pv) :: "memory");
                    ready |= __ballot(pv >= target) << 8;
                    if (!((ready >> (k + 3)) & 1ull)) {
                        const unsigned long long t0 = __builtin_amdgcn_s_memrealtime();
                        for (;;) {
                            unsigned v, to;
                            asm volatile("global_load_dword %0, %2, off sc0 sc1\n\tglobal_load_dword %1, %3, off sc1\n\ts_waitcnt vmcnt(0)"
                                         : "=&v"(v), "=&v"(to) : "v"(cptr), "v"(tmo) : "memory");
                            ready |= __ballot(v >= target) << 8;
                            if ((ready >> (k + 3)) & 1ull) break;
                            if (rfl(to) == call || __builtin_amdgcn_s_memrealtime() - t0 > DEADLINE_TICKS) {
                                asm volatile("global_store_dword %0, %1, off sc1\n\ts_waitcnt vmcnt(0)" :: "v"(tmo), "v"(call) : "memory");
                                ready = ~0ull;
                                break;
                            }
                            __builtin_amdgcn_s_sleep(2);
                        }
                    }
                    if (~ready != 0) {
                        const unsigned S = (unsigned)__builtin_ctzll(~ready);
                        const unsigned km = (S - 5u) - ((S - 5u + 1u - mq) & 3u);
                        ksamp = (int)km >= (int)k ? km : S - 5u;
                        ev |= ((ksamp >= k ? 1ull : 0ull) << ksamp) | (2ull << ksamp);
                    }
                }
            }
            BAR();
        }
        WAITV(0);
        BAR(); BAR(); BAR();
        if (lw == 0 && lane == 0) __hip_atomic_store(callp, call, __ATOMIC_RELAXED, __HIP_MEMORY_SCOPE_AGENT);
        ST_OUT(st_pollk);
    } else {
        const unsigned fr = lane & 15, fq = lane >> 4;
        const unsigned call = GET_CALL();
        unsigned long long evm = 0;
        char* dp = nullptr; char* cp = nullptr;
        const unsigned kc0 = (b & 7u) % PUSH_G, prow = (pq & 3u) * 128u + ((b & 7u) / PUSH_G) * (128u / PUSH_E) + wid * (32u / PUSH_E);
        const int e0 = PUSH_GAP * (int)kc0 - 1;
        constexpr int EP = PUSH_GAP * PUSH_G;
        static_assert(PUSH_GAP >= 2 && PUSH_GAP - 1 + PUSH_D <= 9 && PUSH_GAP <= 7 && PUSH_D < EP && EP - 1 > 4, "push schedule");
        const bool push = pusher;
        const char* const sp = (const char*)p.x + (size_t)(dst * 512u + prow) * 1024u + kc0 * 128u;
        const unsigned vs0 = ((lane >> 3) * 1024u + (((lane & 7) ^ (lane >> 4)) * 16u)) ^ ((prow & 8u) * 8u);
        const unsigned vs1 = PUSH_P > 1 ? (vs0 ^ 64u) + 8192u : vs0 + PUSH_G * 128u;
        constexpr unsigned SO2 = PUSH_P == 4 ? 16384u : PUSH_P == 2 ? PUSH_G * 128u : 2 * PUSH_G * 128u;
#define PUSH_LOAD() asm volatile("global_load_dwordx4 v[232:235], %0, %2 nt\n\tglobal_load_dwordx4 v[236:239], %1, %2 nt\n\t" \
                                 "global_load_dwordx4 v[240:243], %0, %3 nt\n\tglobal_load_dwordx4 v[244:247], %1, %3 nt" \
            :: "v"(vs0), "v"(vs1), "s"(sp), "s"(sp + SO2) : "memory", "v232", "v233", "v234", "v235", "v236", "v237", "v238", "v239", \
               "v240", "v241", "v242", "v243", "v244", "v245", "v246", "v247")
#define ST1(r, ptr, off) asm volatile("global_store_dwordx4 v248, " r ", %0 offset:" #off " sc0 sc1" :: "s"(ptr) : "memory")
        auto push_store = [&](int i) {
            char* const q = dp + i * PUSH_G * 65536u;
            if (PUSH_E == 1) { ST1("v[232:235]", q, 0); ST1("v[236:239]", q, 1024); ST1("v[240:243]", q, 2048); ST1("v[244:247]", q, 3072); }
            else if (PUSH_E == 2) { if (i == 0) { ST1("v[232:235]", q, 0); ST1("v[236:239]", q, 1024); } else { ST1("v[240:243]", q, 0); ST1("v[244:247]", q, 1024); } }
            else { if (i == 0) ST1("v[232:235]", q, 0); else if (i == 1) ST1("v[236:239]", q, 0); else if (i == 2) ST1("v[240:243]", q, 0); else ST1("v[244:247]", q, 0); }
        };
        if (push) {
            char* const pws = p.ws[dst];
            cp = pws + WS_CNT + (rank * 4u + (pq & 3u)) * 128u + kc0 * 4u;
            dp = pws + (call & 1u) * WS_PAR + rank * WS_SRC + kc0 * 65536u + prow * 128u;
            asm volatile("v_mov_b32 v248, %0\n\tv_mov_b32 v249, 0\n\tv_mov_b32 v250, 1" :: "v"(lane * 16u) : "v248", "v249", "v250");
#pragma unroll
            for (int i = 0; i < PUSH_E; ++i) if (e0 + i * EP >= 0) evm |= (1ull << (e0 + i * EP)) | (1ull << (e0 + i * EP + PUSH_D));
            if (kc0 == 0) {
                PUSH_LOAD();
                asm volatile("s_mov_b64 exec, 1\n\tglobal_atomic_add v249, v249, %0 sc1\n\tglobal_atomic_add v249, v249, %1 offset:64 sc1\n\ts_mov_b64 exec, -1\n\t"
                             "s_waitcnt vmcnt(%2)" :: "s"(dp), "s"(cp), "i"(2 + 4 - PUSH_P) : "memory");
                push_store(0);
                evm |= 0x1eu;
            }
        }
        auto push_event = [&](int k) {
            if (!((evm >> k) & 1ull)) return;
            const int d = k - e0, i = d / EP;
            if (d == i * EP) { WAITV(0); push_store(i); }
            else if (k > 4 || kc0 != 0 || k == 4 || __builtin_amdgcn_s_getreg((3 << 11) | 7) == 0) {
                asm volatile("s_waitcnt vmcnt(0)\n\ts_mov_b64 exec, 1\n\tglobal_atomic_add v249, v250, %0 sc1\n\ts_mov_b64 exec, -1" :: "s"(cp + i * PUSH_G * 4u) : "memory");
                evm &= ~0ull << (k < 4 ? 5 : k + 1);
            }
        };
        unsigned a0 = ldsb + A_BASE + 128u * fr + 16u * (fq ^ (fr >> 1));
        unsigned a1 = ldsb + A_BASE + 128u * fr + 16u * ((fq + 4) ^ (fr >> 1));
        const unsigned q = (lane >> 2) & 3, pp = lane & 3;
        const unsigned fB = (q & 2) | ((fq & 1) << 2);
        unsigned bb[4], bh[4];
#pragma unroll
        for (int c = 0; c < 4; ++c) {
            bb[c] = ldsb + 512u * (8 * fq + q) + 128u * (wid ^ (q & 1)) + 16u * ((2 * c + (pp >> 1)) ^ fB) + 8u * (pp & 1);
            bh[c] = bb[c] + 2 * B_SLOT;
        }
        f32x4 acc[8][4];
        bf16x8 X0[8], X1[8];

        s16x4 Wl0[4], Wh0[4], Wl1[4], Wh1[4];
#define WFRAG(S, c) __builtin_bit_cast(bf16x8, __builtin_shufflevector(Wl##S[c], Wh##S[c], 0, 1, 2, 3, 4, 5, 6, 7))
#define MF(S, i) do { const int m_ = (i) >> 2, c_ = (m_ & 1) ? 3 - ((i) & 3) : ((i) & 3); \
            asm volatile("v_mfma_f32_16x16x32_bf16 %0, %1, %2, %0" : "+v"(acc[m_][c_]) : "v"(WFRAG(S, c_)), "v"(X##S[m_]) : "memory"); } while (0)
#define MF0(S, i) do { const int m_ = (i) >> 2, c_ = (m_ & 1) ? 3 - ((i) & 3) : ((i) & 3); \
            asm volatile("v_mfma_f32_16x16x32_bf16 %0, %1, %2, 0" : "=&v"(acc[m_][c_]) : "v"(WFRAG(S, c_)), "v"(X##S[m_]) : "memory"); } while (0)
#define RD(S, r, j) do { if ((j) < 8) { const unsigned o_ = ((r) == 2 ? bh[((j) >> 1) & 3] : bb[((j) >> 1) & 3] + (r) * B_SLOT) + ((S) * 4096u + ((j) & 1) * 512u) * (WLAY == 1 ? 4u : 1u); \
                if ((j) & 1) Wh##S[((j) >> 1) & 3] = __builtin_amdgcn_ds_read_tr16_b64_v4i16((LDS s16x4*)o_); \
                else         Wl##S[((j) >> 1) & 3] = __builtin_amdgcn_ds_read_tr16_b64_v4i16((LDS s16x4*)o_); } \
            else X##S[(j) & 7] = *(const LDS bf16x8*)(((S) ? a1 : a0) + (r) * A_SLOT + 2048u * ((j) & 7)); } while (0)
#define RUN28(MFX, SM, i0, SR, r) do { _Pragma("unroll") for (int g = 0; g < 4; ++g) { \
            MFX(SM, (i0) + 7 * g); RD(SR, r, 4 * g); MFX(SM, (i0) + 7 * g + 1); MFX(SM, (i0) + 7 * g + 2); RD(SR, r, 4 * g + 1); \
            MFX(SM, (i0) + 7 * g + 3); MFX(SM, (i0) + 7 * g + 4); RD(SR, r, 4 * g + 2); MFX(SM, (i0) + 7 * g + 5); MFX(SM, (i0) + 7 * g + 6); RD(SR, r, 4 * g + 3); } } while (0)
#define RUN4(MFX, SM, i0) do { MFX(SM, i0); MFX(SM, (i0) + 1); MFX(SM, (i0) + 2); MFX(SM, (i0) + 3); } while (0)
#define TO_BARRIER() do { __builtin_amdgcn_s_waitcnt(0xC07F); ST_BEG(); BAR(); ST_END(st_bar); } while (0)
#define EVPOS 2
#define BODY(r, i) do { RUN28(MF, 1, 4, 0, r); RUN28(MF, 0, 0, 1, r); RUN4(MF, 0, 28); \
            _Pragma("unroll") for (int j = 0; j < EVPOS; ++j) MF(1, j); \
            if (__builtin_expect((em >> (i)) & 1u, 0)) push_event(kb + (i)); \
            _Pragma("unroll") for (int j = EVPOS; j < 4; ++j) MF(1, j); TO_BARRIER(); } while (0)

        asm volatile("" : "+v"(a0), "+v"(a1), "+v"(bb[0]), "+v"(bb[1]), "+v"(bb[2]), "+v"(bb[3]), "+v"(bh[0]), "+v"(bh[1]), "+v"(bh[2]), "+v"(bh[3]));
        BAR();
        if (push && kc0 != 0) PUSH_LOAD();
        ST_FIRST();
#pragma unroll
        for (int j = 0; j < 16; ++j) RD(0, 0, j);
        RUN28(MF0, 0, 0, 1, 0); RUN4(MF0, 0, 28); RUN4(MF, 1, 0);
        TO_BARRIER();
        for (int kb = 1; kb < NSTEP; kb += 3) {
            unsigned em = (unsigned)(evm >> kb);
            asm volatile("" : "+s"(em));
            BODY(1, 0); BODY(2, 1); BODY(0, 2);
        }
        const unsigned tv = __hip_atomic_load(tmo, __ATOMIC_RELAXED, __HIP_MEMORY_SCOPE_AGENT);
#pragma unroll
        for (int i = 4; i < 32; ++i) MF(1, i);
        asm volatile("s_nop 7\n\ts_nop 1" ::: "memory");
        SB0();
        const bool bad = tv == call;
        char* orow = (char*)p.out + (size_t)(mq * 128 + fr) * WROW + colb + wid * 128u + (16 * (fq & 1) + 8 * (fq >> 1)) * 2;
#pragma unroll
        for (int m = 0; m < 8; ++m) {
            u32x4 o[2];
#pragma unroll
            for (int h = 0; h < 2; ++h) {
                const f32x4 u = acc[m][2 * h], v = acc[m][2 * h + 1];
                const unsigned u0 = pack_relu(u[0], u[1]), u1 = pack_relu(u[2], u[3]);
                const unsigned v0 = pack_relu(v[0], v[1]), v1 = pack_relu(v[2], v[3]);
                const auto s0 = __builtin_amdgcn_permlane16_swap(u0, v0, false, false);
                const auto s1 = __builtin_amdgcn_permlane16_swap(u1, v1, false, false);
                o[h] = (u32x4){s0[0], s1[0], s0[1], s1[1]};
                if (bad) o[h] = (u32x4){0x7fc07fc0u, 0x7fc07fc0u, 0x7fc07fc0u, 0x7fc07fc0u};
            }
            asm volatile("global_store_dwordx4 %0, %1, off sc0 sc1\n\tglobal_store_dwordx4 %0, %2, off offset:64 sc0 sc1\n\ts_nop 1"
                         :: "v"(orow + (size_t)m * 16 * WROW), "v"(o[0]), "v"(o[1]) : "memory");
        }
        BAR();
    }
}

}

extern "C" void kernel_launch(int rank, int nranks, ncclComm_t comm,
                              hipStream_t stream,
                              void* const* d_in, const int* in_sizes, int n_in,
                              void* d_out, int out_size,
                              const int* in_dtypes, int out_dtype,
                              void* const* d_out_all,
                              void* const* d_ws_all, size_t ws_bytes) {
    using namespace a2ag;
    if (nranks != NR || (unsigned)rank >= (unsigned)NR || n_in < 2 || ws_bytes < WS_END ||
        in_sizes[0] != 4096 * KSH || in_sizes[1] != KTOT * NTOT || out_size != MLOC * NTOT) {
        fprintf(stderr, "kernel_launch: unexpected shapes\n");
        return;
    }
    HIP_CHECK(hipFuncSetAttribute((const void*)a2a_gemm, hipFuncAttributeMaxDynamicSharedMemorySize, (int)LDS_BYTES));
    Params p;
    p.x = d_in[0]; p.w = d_in[1]; p.out = d_out; p.myws = (char*)d_ws_all[rank];
    for (int i = 0; i < NR; ++i) p.ws[i] = (char*)d_ws_all[i];
    p.rank = rank; p.pad = 0;
    hipLaunchKernelGGL(a2a_gemm, dim3(NWG), dim3(512), LDS_BYTES, stream, p);
    HIP_CHECK(hipGetLastError());
}
```

```cpp
#include <hip/hip_runtime.h>
#include <rccl/rccl.h>
#include <cstdio>

#define HIP_CHECK(expr) do { hipError_t _e = (expr); if (_e != hipSuccess) { \
    fprintf(stderr, "HIP error %s at %s:%d\n", hipGetErrorString(_e), __FILE__, __LINE__); \
    return; } } while (0)

namespace a2ag {

typedef __bf16 bf16x8 __attribute__((ext_vector_type(8)));
typedef __bf16 bf16x2 __attribute__((ext_vector_type(2)));
typedef short s16x4 __attribute__((ext_vector_type(4)));
typedef float f32x2 __attribute__((ext_vector_type(2)));
typedef float f32x4 __attribute__((ext_vector_type(4)));
typedef unsigned u32x4 __attribute__((ext_vector_type(4)));
#define LDS __attribute__((address_space(3)))
#define DI __device__ __forceinline__

constexpr int NR = 8;
constexpr int MLOC = 512;
constexpr int KSH = 512;
constexpr int KTOT = 4096, NTOT = 16384;
constexpr int NSTEP = KTOT / 64;
constexpr int NWG = 256;
constexpr unsigned B_SLOT = 32768, A_SLOT = 16384;
constexpr unsigned A_BASE = 3 * B_SLOT;
constexpr unsigned LDS_BYTES = A_BASE + 3 * A_SLOT;
constexpr unsigned X_BYTES = 4096u * KSH * 2u;
constexpr unsigned W_BYTES = (unsigned)KTOT * NTOT * 2u;
constexpr unsigned WROW = NTOT * 2u;

constexpr unsigned WS_SRC = 512u * 1024u;
constexpr unsigned WS_PAR = NR * WS_SRC;
constexpr unsigned WS_CNT = 2 * WS_PAR;
constexpr unsigned WS_CALL = WS_CNT + NR * 4 * 128;
constexpr unsigned WS_TMO = WS_CALL + NWG * 128;
constexpr unsigned WS_END = WS_TMO + 128;
#define PUSH_E 4
constexpr unsigned PUSH_G = 8 / PUSH_E, PUSH_P = 4 / PUSH_E;
constexpr unsigned CNT_PER_CALL = 4 * PUSH_E;
constexpr unsigned long long DEADLINE_TICKS = 50000000ull;

#define PUSH_GAP 3
#define PUSH_D 5
#define WLAY 1
#define WPOL 0
#define PACE 85
#define PACE_N 30
#define PACE2 71
#define PACE_T0 280
#define KORD 2

#define ST_DECL
#define ST_BEG()
#define ST_END(acc)
#define ST_FIRST()
#define ST_OUT(x6)

struct Params {
    const void* x; const void* w; void* out; char* myws;
    char* ws[NR];
    int rank; int pad;
};

#define BAR() do { asm volatile("" ::: "memory"); __builtin_amdgcn_s_barrier(); asm volatile("" ::: "memory"); } while (0)
#define WAITV(n) asm volatile("s_waitcnt vmcnt(" #n ")" ::: "memory")
#define SB0() __builtin_amdgcn_sched_barrier(0)

DI unsigned rfl(unsigned v) { return __builtin_amdgcn_readfirstlane(v); }

#define DMA4(suffix, lds, v, srd, soff) asm volatile("s_mov_b32 m0, %0\n\ts_nop 0\n\t" \
        "buffer_load_dwordx4 %1, %5, %6 offen " suffix " lds\n\t" \
        "buffer_load_dwordx4 %2, %5, %6 offen offset:1024 " suffix " lds\n\t" \
        "buffer_load_dwordx4 %3, %5, %6 offen offset:2048 " suffix " lds\n\t" \
        "buffer_load_dwordx4 %4, %5, %6 offen offset:3072 " suffix " lds" \
        :: "s"(lds), "v"((v)[0]), "v"((v)[1]), "v"((v)[2]), "v"((v)[3]), "s"(srd), "s"(soff) : "memory")

DI void step_map(unsigned t, unsigned& j, unsigned& kc) {
    if (t < 8) { j = 0; kc = t; }
    else { const unsigned u = t - 8; kc = (u * 147u) >> 10; j = 1 + (u - 7 * kc); }
}

DI unsigned stage_map(unsigned s, unsigned q) {
    const unsigned o = (q + 2u) & 3u, v = (s - o) & 3u, s0 = s - v;
    if (s < o || s0 + 3u >= NSTEP || (s0 < 8u && s0 + 2u >= 8u)) return s;
    return v == 0 ? s + 2u : (v == 3 ? s : s - 1u);
}

DI unsigned pack_relu(float a, float b) {
    f32x2 v = {__builtin_elementwise_maximum(a, 0.f), __builtin_elementwise_maximum(b, 0.f)};
    bf16x2 r = __builtin_convertvector(v, bf16x2);
    return __builtin_bit_cast(unsigned, r);
}

__global__ __launch_bounds__(512, 2) __attribute__((amdgpu_num_vgpr(232))) void a2a_gemm(Params p) {
    extern __shared__ __attribute__((aligned(16))) char smem[];
    const unsigned ldsb = (unsigned)(size_t)(LDS char*)smem;
    const unsigned tid = threadIdx.x, lane = tid & 63;
    const unsigned wid = rfl(tid >> 6);
    const unsigned b = blockIdx.x;
    const unsigned mq = (b >> 3) & 3;
    const unsigned colb = (b & 7) * 4096u + (b >> 5) * 512u;
    const unsigned rank = (unsigned)p.rank & (NR - 1);
    const unsigned pi = (b >> 3) - (b >> 6), pq = (pi * 37u) >> 8;
    const bool pusher = ((b >> 3) & 7u) != 7u;
    const unsigned dst = (rank + 1u + (pi - pq * 7u)) & (NR - 1);
    char* const myws = p.myws;
    unsigned* const callp = (unsigned*)(myws + WS_CALL + b * 128);
    unsigned* const tmo = (unsigned*)(myws + WS_TMO);
#define GET_CALL() (rfl(*(const volatile unsigned __attribute__((address_space(4)))*)(size_t)callp) + 1u)
    ST_DECL;

    if (wid >= 4) {
        const unsigned lw = wid - 4;
        const unsigned tstart = (unsigned)__builtin_amdgcn_s_memrealtime();
        const __amdgpu_buffer_rsrc_t srdW = __builtin_amdgcn_make_buffer_rsrc((void*)p.w, (short)0, (int)W_BYTES, 0x00020000);
        const __amdgpu_buffer_rsrc_t srdX = __builtin_amdgcn_make_buffer_rsrc((void*)p.x, (short)0, (int)X_BYTES, 0x00020000);
        const __amdgpu_buffer_rsrc_t srdR = __builtin_amdgcn_make_buffer_rsrc((void*)myws, (short)0, (int)WS_CNT, 0x00020000);
        unsigned vW[4], vW2[4], vL[4];
        const unsigned vR = lane * 16u;
#pragma unroll
        for (int j = 0; j < 4; ++j) {
            vW[j] = (lane >> 5) * WROW + ((((lane >> 3) & 3) ^ (lane >> 5)) * 128u) + (((lane & 7) ^ ((j & 1) << 1)) * 16u) + j * (2u * WROW - 1024u);
            vW2[j] = vW[j] ^ 64u;
            vL[j] = (lane >> 3) * 1024u + (((lane & 7) ^ (4 * (j & 1) + (lane >> 4))) * 16u) + j * (8192u - 1024u);
        }
        const unsigned sW = colb + lw * 16u * WROW;
        constexpr unsigned WQ2 = 8u * WROW;
        const unsigned lW = ldsb + lw * 8192u;
        const unsigned sL = (rank * 512u + mq * 128u + lw * 32u) * 1024u;
        const unsigned lA = ldsb + A_BASE + lw * 4096u;
        unsigned tW, tA;
        { unsigned j, kc; step_map(stage_map(lane, mq), j, kc);
          const unsigned src = (rank + j) & (NR - 1);
          tW = sW + (src * KSH + kc * 64u) * WROW;
          tA = j == 0 ? sL + 128u * kc : mq * 16384u + lw * 4096u + src * WS_SRC + 65536u * kc; }
#define RL(t, s) __builtin_amdgcn_readlane(t, s)
#define ISSUE_W(mw, ow) do { DMA4("", mw, vW, srdW, ow); DMA4("", (mw) + 4096u, vW2, srdW, (ow) + WQ2); } while (0)
        __amdgpu_buffer_rsrc_t srdA = srdX;
        unsigned vA[4] = {vL[0], vL[1], vL[2], vL[3]};
#define ISSUE(mw, ow, ma, oa) do { ISSUE_W(mw, ow); DMA4("sc0 sc1", ma, vA, srdA, oa); } while (0)
        { const unsigned w0 = RL(tW, 0), a0 = RL(tA, 0), w1 = RL(tW, 1), a1 = RL(tA, 1);
          asm volatile("s_nop 4" ::: "memory");
          ISSUE(lW, w0, lA, a0); ISSUE(lW + B_SLOT, w1, lA + A_SLOT, a1); }
        unsigned ow = RL(tW, 2), oa = RL(tA, 2);
        const unsigned call = GET_CALL();
        WAITV(12);
        BAR();
        ISSUE(lW + 2 * B_SLOT, ow, lA + 2 * A_SLOT, oa);
        ow = RL(tW, 3); oa = RL(tA, 3);
        if (lw == 3) {
            const unsigned vp = (rank * 512u + mq * 128u + (b >> 5) * 16u + (lane & 15)) * 1024u + (4u + (lane >> 4)) * 128u;
            asm volatile("buffer_load_dword v220, %0, %1, 0 offen" :: "v"(vp), "s"(srdX) : "memory", "v220");
        }
        if (lane >= 8) tA += (call & 1u) * WS_PAR;
        const unsigned target = call * CNT_PER_CALL;
        unsigned long long ready = 0xffull;
        const unsigned* cptr;
        { unsigned j, kc; step_map(stage_map(8u + (lane < 56 ? lane : 55u), mq), j, kc);
          cptr = (const unsigned*)(myws + WS_CNT + (((rank + j) & (NR - 1)) * 4u + mq) * 128u + kc * 4u); }
        unsigned ksamp = 3;
        unsigned long long ev = (1ull << 61) | (1ull << 5);
        if (lw == 0 && ~ready != 0) ev |= 3ull << 3;
        unsigned mw = lW, ma = lA, k = 0;
#define ST_LINE()
#define STEP() do { ISSUE(mw, ow, ma, oa); ow = RL(tW, k + 4); oa = RL(tA, k + 4); ST_LINE(); \
            mw = (mw == lW + 2 * B_SLOT) ? lW : mw + B_SLOT; ma = (ma == lA + 2 * A_SLOT) ? lA : ma + A_SLOT; ++k; \
            asm volatile("" : "+s"(ow), "+s"(oa), "+s"(mw), "+s"(ma), "+s"(k)); } while (0)
        asm volatile("" : "+s"(ow), "+s"(oa), "+s"(mw), "+s"(ma), "+s"(k));
        WAITV(12);
        const unsigned tp0 = tstart + PACE_T0;
        BAR();
        for (;;) {
            const unsigned kend = rfl(k + (unsigned)__builtin_ctzll(ev >> k));
            if (lw == 3) while (k != kend) {
                STEP(); WAITV(12);
                { const unsigned due = tp0 + k * PACE - (k > PACE_N ? (k - PACE_N) * (PACE - PACE2) : 0u); while ((int)((unsigned)__builtin_amdgcn_s_memrealtime() - due) < 0) __builtin_amdgcn_s_sleep(1); }
                BAR();
            }
            while (k != kend) { STEP(); WAITV(12); BAR(); }
            if (k == NSTEP - 3) break;
            ev &= ~(1ull << k);
            if (k == 5) { srdA = srdR; vA[0] = vA[1] = vA[2] = vA[3] = vR; }
            if (lw == 0 && k == ksamp) {
                asm volatile("global_load_dword v221, %0, off sc0 sc1" :: "v"(cptr) : "memory", "v221");
                STEP(); WAITV(13);
            } else {
                STEP(); WAITV(12);
                if (lw == 0 && k == ksamp + 2) {
                    unsigned pv; asm volatile("v_mov_b32 %0, v221" : "=v"(pv) :: "memory");
                    ready |= __ballot(pv >= target) << 8;
                    if (!((ready >> (k + 3)) & 1ull)) {
                        const unsigned long long t0 = __builtin_amdgcn_s_memrealtime();
                        for (;;) {
                            unsigned v, to;
                            asm volatile("global_load_dword %0, %2, off sc0 sc1\n\tglobal_load_dword %1, %3, off sc1\n\ts_waitcnt vmcnt(0)"
                                         : "=&v"(v), "=&v"(to) : "v"(cptr), "v"(tmo) : "memory");
                            ready |= __ballot(v >= target) << 8;
                            if ((ready >> (k + 3)) & 1ull) break;
                            if (rfl(to) == call || __builtin_amdgcn_s_memrealtime() - t0 > DEADLINE_TICKS) {
                                asm volatile("global_store_dword %0, %1, off sc1\n\ts_waitcnt vmcnt(0)" :: "v"(tmo), "v"(call) : "memory");
                                ready = ~0ull;
                                break;
                            }
                            __builtin_amdgcn_s_sleep(2);
                        }
                    }
                    if (~ready != 0) {
                        const unsigned S = (unsigned)__builtin_ctzll(~ready);
                        const unsigned km = (S - 5u) - ((S - 5u + 1u - mq) & 3u);
                        ksamp = (int)km >= (int)k ? km : S - 5u;
                        ev |= ((ksamp >= k ? 1ull : 0ull) << ksamp) | (2ull << ksamp);
                    }
                }
            }
            BAR();
        }
        WAITV(0);
        BAR(); BAR(); BAR();
        if (lw == 0 && lane == 0) __hip_atomic_store(callp, call, __ATOMIC_RELAXED, __HIP_MEMORY_SCOPE_AGENT);
        ST_OUT(st_pollk);
    } else {
        const unsigned fr = lane & 15, fq = lane >> 4;
        const unsigned call = GET_CALL();
        unsigned long long evm = 0;
        char* dp = nullptr; char* cp = nullptr;
        const unsigned kc0 = (b & 7u) % PUSH_G, prow = (pq & 3u) * 128u + ((b & 7u) / PUSH_G) * (128u / PUSH_E) + wid * (32u / PUSH_E);
        const int e0 = PUSH_GAP * (int)kc0 - 1;
        constexpr int EP = PUSH_GAP * PUSH_G;
        static_assert(PUSH_GAP >= 2 && PUSH_GAP - 1 + PUSH_D <= 9 && PUSH_GAP <= 7 && PUSH_D < EP && EP - 1 > 4, "push schedule");
        const bool push = pusher;
        const char* const sp = (const char*)p.x + (size_t)(dst * 512u + prow) * 1024u + kc0 * 128u;
        const unsigned vs0 = ((lane >> 3) * 1024u + (((lane & 7) ^ (lane >> 4)) * 16u)) ^ ((prow & 8u) * 8u);
        const unsigned vs1 = PUSH_P > 1 ? (vs0 ^ 64u) + 8192u : vs0 + PUSH_G * 128u;
        constexpr unsigned SO2 = PUSH_P == 4 ? 16384u : PUSH_P == 2 ? PUSH_G * 128u : 2 * PUSH_G * 128u;
#define PUSH_LOAD() asm volatile("global_load_dwordx4 v[232:235], %0, %2 nt\n\tglobal_load_dwordx4 v[236:239], %1, %2 nt\n\t" \
                                 "global_load_dwordx4 v[240:243], %0, %3 nt\n\tglobal_load_dwordx4 v[244:247], %1, %3 nt" \
            :: "v"(vs0), "v"(vs1), "s"(sp), "s"(sp + SO2) : "memory", "v232", "v233", "v234", "v235", "v236", "v237", "v238", "v239", \
               "v240", "v241", "v242", "v243", "v244", "v245", "v246", "v247")
#define ST1(r, ptr, off) asm volatile("global_store_dwordx4 v248, " r ", %0 offset:" #off " sc0 sc1" :: "s"(ptr) : "memory")
        auto push_store = [&](int i) {
            char* const q = dp + i * PUSH_G * 65536u;
            if (PUSH_E == 1) { ST1("v[232:235]", q, 0); ST1("v[236:239]", q, 1024); ST1("v[240:243]", q, 2048); ST1("v[244:247]", q, 3072); }
            else if (PUSH_E == 2) { if (i == 0) { ST1("v[232:235]", q, 0); ST1("v[236:239]", q, 1024); } else { ST1("v[240:243]", q, 0); ST1("v[244:247]", q, 1024); } }
            else { if (i == 0) ST1("v[232:235]", q, 0); else if (i == 1) ST1("v[236:239]", q, 0); else if (i == 2) ST1("v[240:243]", q, 0); else ST1("v[244:247]", q, 0); }
        };
        if (push) {
            char* const pws = p.ws[dst];
            cp = pws + WS_CNT + (rank * 4u + (pq & 3u)) * 128u + kc0 * 4u;
            dp = pws + (call & 1u) * WS_PAR + rank * WS_SRC + kc0 * 65536u + prow * 128u;
            asm volatile("v_mov_b32 v248, %0\n\tv_mov_b32 v249, 0\n\tv_mov_b32 v250, 1" :: "v"(lane * 16u) : "v248", "v249", "v250");
#pragma unroll
            for (int i = 0; i < PUSH_E; ++i) if (e0 + i * EP >= 0) evm |= (1ull << (e0 + i * EP)) | (1ull << (e0 + i * EP + PUSH_D));
            if (kc0 == 0) {
                PUSH_LOAD();
                asm volatile("s_mov_b64 exec, 1\n\tglobal_atomic_add v249, v249, %0 sc1\n\tglobal_atomic_add v249, v249, %1 offset:64 sc1\n\ts_mov_b64 exec, -1\n\t"
                             "s_waitcnt vmcnt(%2)" :: "s"(dp), "s"(cp), "i"(2 + 4 - PUSH_P) : "memory");
                push_store(0);
                evm |= 0x1eu;
            }
        }
        auto push_event = [&](int k) {
            if (!((evm >> k) & 1ull)) return;
            const int d = k - e0, i = d / EP;
            if (d == i * EP) { WAITV(0); push_store(i); }
            else if (k > 4 || kc0 != 0 || k == 4 || __builtin_amdgcn_s_getreg((3 << 11) | 7) == 0) {
                asm volatile("s_waitcnt vmcnt(0)\n\ts_mov_b64 exec, 1\n\tglobal_atomic_add v249, v250, %0 sc1\n\ts_mov_b64 exec, -1" :: "s"(cp + i * PUSH_G * 4u) : "memory");
                evm &= ~0ull << (k < 4 ? 5 : k + 1);
            }
        };
        unsigned a0 = ldsb + A_BASE + 128u * fr + 16u * (fq ^ (fr >> 1));
        unsigned a1 = ldsb + A_BASE + 128u * fr + 16u * ((fq + 4) ^ (fr >> 1));
        const unsigned q = (lane >> 2) & 3, pp = lane & 3;
        const unsigned fB = (q & 2) | ((fq & 1) << 2);
        unsigned bb[4], bh[4];
#pragma unroll
        for (int c = 0; c < 4; ++c) {
            bb[c] = ldsb + 512u * (8 * fq + q) + 128u * (wid ^ (q & 1)) + 16u * ((2 * c + (pp >> 1)) ^ fB) + 8u * (pp & 1);
            bh[c] = bb[c] + 2 * B_SLOT;
        }
        f32x4 acc[8][4];
        bf16x8 X0[8], X1[8];

        s16x4 Wl0[4], Wh0[4], Wl1[4], Wh1[4];
#define WFRAG(S, c) __builtin_bit_cast(bf16x8, __builtin_shufflevector(Wl##S[c], Wh##S[c], 0, 1, 2, 3, 4, 5, 6, 7))
#define MF(S, i) do { const int m_ = (i) >> 2, c_ = (m_ & 1) ? 3 - ((i) & 3) : ((i) & 3); \
            asm volatile("v_mfma_f32_16x16x32_bf16 %0, %1, %2, %0" : "+v"(acc[m_][c_]) : "v"(WFRAG(S, c_)), "v"(X##S[m_]) : "memory"); } while (0)
#define MF0(S, i) do { const int m_ = (i) >> 2, c_ = (m_ & 1) ? 3 - ((i) & 3) : ((i) & 3); \
            asm volatile("v_mfma_f32_16x16x32_bf16 %0, %1, %2, 0" : "=&v"(acc[m_][c_]) : "v"(WFRAG(S, c_)), "v"(X##S[m_]) : "memory"); } while (0)
#define RD(S, r, j) do { if ((j) < 8) { const unsigned o_ = ((r) == 2 ? bh[((j) >> 1) & 3] : bb[((j) >> 1) & 3] + (r) * B_SLOT) + ((S) * 4096u + ((j) & 1) * 512u) * (WLAY == 1 ? 4u : 1u); \
                if ((j) & 1) Wh##S[((j) >> 1) & 3] = __builtin_amdgcn_ds_read_tr16_b64_v4i16((LDS s16x4*)o_); \
                else         Wl##S[((j) >> 1) & 3] = __builtin_amdgcn_ds_read_tr16_b64_v4i16((LDS s16x4*)o_); } \
            else X##S[(j) & 7] = *(const LDS bf16x8*)(((S) ? a1 : a0) + (r) * A_SLOT + 2048u * ((j) & 7)); } while (0)
#define RUN28(MFX, SM, i0, SR, r) do { _Pragma("unroll") for (int g = 0; g < 4; ++g) { \
            MFX(SM, (i0) + 7 * g); RD(SR, r, 4 * g); MFX(SM, (i0) + 7 * g + 1); MFX(SM, (i0) + 7 * g + 2); RD(SR, r, 4 * g + 1); \
            MFX(SM, (i0) + 7 * g + 3); MFX(SM, (i0) + 7 * g + 4); RD(SR, r, 4 * g + 2); MFX(SM, (i0) + 7 * g + 5); MFX(SM, (i0) + 7 * g + 6); RD(SR, r, 4 * g + 3); } } while (0)
#define RUN4(MFX, SM, i0) do { MFX(SM, i0); MFX(SM, (i0) + 1); MFX(SM, (i0) + 2); MFX(SM, (i0) + 3); } while (0)
#define TO_BARRIER() do { __builtin_amdgcn_s_waitcnt(0xC07F); ST_BEG(); BAR(); ST_END(st_bar); } while (0)
#define EVPOS 2
#define BODY(r, i) do { RUN28(MF, 1, 4, 0, r); RUN28(MF, 0, 0, 1, r); RUN4(MF, 0, 28); \
            _Pragma("unroll") for (int j = 0; j < EVPOS; ++j) MF(1, j); \
            if (__builtin_expect((em >> (i)) & 1u, 0)) push_event(kb + (i)); \
            _Pragma("unroll") for (int j = EVPOS; j < 4; ++j) MF(1, j); TO_BARRIER(); } while (0)

        asm volatile("" : "+v"(a0), "+v"(a1), "+v"(bb[0]), "+v"(bb[1]), "+v"(bb[2]), "+v"(bb[3]), "+v"(bh[0]), "+v"(bh[1]), "+v"(bh[2]), "+v"(bh[3]));
        BAR();
        if (push && kc0 != 0) PUSH_LOAD();
        ST_FIRST();
#pragma unroll
        for (int j = 0; j < 16; ++j) RD(0, 0, j);
        RUN28(MF0, 0, 0, 1, 0); RUN4(MF0, 0, 28); RUN4(MF, 1, 0);
        TO_BARRIER();
        for (int kb = 1; kb < NSTEP; kb += 3) {
            unsigned em = (unsigned)(evm >> kb);
            asm volatile("" : "+s"(em));
            BODY(1, 0); BODY(2, 1); BODY(0, 2);
        }
        const unsigned tv = __hip_atomic_load(tmo, __ATOMIC_RELAXED, __HIP_MEMORY_SCOPE_AGENT);
#pragma unroll
        for (int i = 4; i < 32; ++i) MF(1, i);
        asm volatile("s_nop 7\n\ts_nop 1" ::: "memory");
        SB0();
        const bool bad = tv == call;
        char* orow = (char*)p.out + (size_t)(mq * 128 + fr) * WROW + colb + wid * 128u + (16 * (fq & 1) + 8 * (fq >> 1)) * 2;
#pragma unroll
        for (int m = 0; m < 8; ++m) {
            u32x4 o[2];
#pragma unroll
            for (int h = 0; h < 2; ++h) {
                const f32x4 u = acc[m][2 * h], v = acc[m][2 * h + 1];
                const unsigned u0 = pack_relu(u[0], u[1]), u1 = pack_relu(u[2], u[3]);
                const unsigned v0 = pack_relu(v[0], v[1]), v1 = pack_relu(v[2], v[3]);
                const auto s0 = __builtin_amdgcn_permlane16_swap(u0, v0, false, false);
                const auto s1 = __builtin_amdgcn_permlane16_swap(u1, v1, false, false);
                o[h] = (u32x4){s0[0], s1[0], s0[1], s1[1]};
                if (bad) o[h] = (u32x4){0x7fc07fc0u, 0x7fc07fc0u, 0x7fc07fc0u, 0x7fc07fc0u};
            }
            asm volatile("global_store_dwordx4 %0, %1, off sc0 sc1\n\tglobal_store_dwordx4 %0, %2, off offset:64 sc0 sc1\n\ts_nop 1"
                         :: "v"(orow + (size_t)m * 16 * WROW), "v"(o[0]), "v"(o[1]) : "memory");
        }
        BAR();
    }
}

}

extern "C" void kernel_launch(int rank, int nranks, ncclComm_t comm,
                              hipStream_t stream,
                              void* const* d_in, const int* in_sizes, int n_in,
                              void* d_out, int out_size,
                              const int* in_dtypes, int out_dtype,
                              void* const* d_out_all,
                              void* const* d_ws_all, size_t ws_bytes) {
    using namespace a2ag;
    if (nranks != NR || (unsigned)rank >= (unsigned)NR || n_in < 2 || ws_bytes < WS_END ||
        in_sizes[0] != 4096 * KSH || in_sizes[1] != KTOT * NTOT || out_size != MLOC * NTOT) {
        fprintf(stderr, "kernel_launch: unexpected shapes\n");
        return;
    }
    HIP_CHECK(hipFuncSetAttribute((const void*)a2a_gemm, hipFuncAttributeMaxDynamicSharedMemorySize, (int)LDS_BYTES));
    Params p;
    p.x = d_in[0]; p.w = d_in[1]; p.out = d_out; p.myws = (char*)d_ws_all[rank];
    for (int i = 0; i < NR; ++i) p.ws[i] = (char*)d_ws_all[i];
    p.rank = rank; p.pad = 0;
    hipLaunchKernelGGL(a2a_gemm, dim3(NWG), dim3(512), LDS_BYTES, stream, p);
    HIP_CHECK(hipGetLastError());
}
```

```cpp
#include <hip/hip_runtime.h>
#include <rccl/rccl.h>
#include <cstdio>

#define HIP_CHECK(expr) do { hipError_t _e = (expr); if (_e != hipSuccess) { \
    fprintf(stderr, "HIP error %s at %s:%d\n", hipGetErrorString(_e), __FILE__, __LINE__); \
    return; } } while (0)

namespace a2ag {

typedef __bf16 bf16x8 __attribute__((ext_vector_type(8)));
typedef __bf16 bf16x2 __attribute__((ext_vector_type(2)));
typedef short s16x4 __attribute__((ext_vector_type(4)));
typedef float f32x2 __attribute__((ext_vector_type(2)));
typedef float f32x4 __attribute__((ext_vector_type(4)));
typedef unsigned u32x4 __attribute__((ext_vector_type(4)));
#define LDS __attribute__((address_space(3)))
#define DI __device__ __forceinline__

constexpr int NR = 8;
constexpr int MLOC = 512;
constexpr int KSH = 512;
constexpr int KTOT = 4096, NTOT = 16384;
constexpr int NSTEP = KTOT / 64;
constexpr int NWG = 256;
constexpr unsigned B_SLOT = 32768, A_SLOT = 16384;
constexpr unsigned A_BASE = 3 * B_SLOT;
constexpr unsigned LDS_BYTES = A_BASE + 3 * A_SLOT;
constexpr unsigned X_BYTES = 4096u * KSH * 2u;
constexpr unsigned W_BYTES = (unsigned)KTOT * NTOT * 2u;
constexpr unsigned WROW = NTOT * 2u;

constexpr unsigned WS_SRC = 512u * 1024u;
constexpr unsigned WS_PAR = NR * WS_SRC;
constexpr unsigned WS_CNT = 2 * WS_PAR;
constexpr unsigned WS_CALL = WS_CNT + NR * 4 * 128;
constexpr unsigned WS_TMO = WS_CALL + NWG * 128;
constexpr unsigned WS_END = WS_TMO + 128;
#define PUSH_E 4
constexpr unsigned PUSH_G = 8 / PUSH_E, PUSH_P = 4 / PUSH_E;
constexpr unsigned CNT_PER_CALL = 4 * PUSH_E;
constexpr unsigned long long DEADLINE_TICKS = 50000000ull;

#define PUSH_GAP 3
#define PUSH_D 5
#define WLAY 1
#define WPOL 0
#define PACE 85
#define PACE_N 30
#define PACE2 71
#define PACE_T0 280
#define KORD 2

#define ST_DECL
#define ST_BEG()
#define ST_END(acc)
#define ST_FIRST()
#define ST_OUT(x6)

struct Params {
    const void* x; const void* w; void* out; char* myws;
    char* ws[NR];
    int rank; int pad;
};

#define BAR() do { asm volatile("" ::: "memory"); __builtin_amdgcn_s_barrier(); asm volatile("" ::: "memory"); } while (0)
#define WAITV(n) asm volatile("s_waitcnt vmcnt(" #n ")" ::: "memory")
#define SB0() __builtin_amdgcn_sched_barrier(0)

DI unsigned rfl(unsigned v) { return __builtin_amdgcn_readfirstlane(v); }

#define DMA4(suffix, lds, v, srd, soff) asm volatile("s_mov_b32 m0, %0\n\ts_nop 0\n\t" \
        "buffer_load_dwordx4 %1, %5, %6 offen " suffix " lds\n\t" \
        "buffer_load_dwordx4 %2, %5, %6 offen offset:1024 " suffix " lds\n\t" \
        "buffer_load_dwordx4 %3, %5, %6 offen offset:2048 " suffix " lds\n\t" \
        "buffer_load_dwordx4 %4, %5, %6 offen offset:3072 " suffix " lds" \
        :: "s"(lds), "v"((v)[0]), "v"((v)[1]), "v"((v)[2]), "v"((v)[3]), "s"(srd), "s"(soff) : "memory")

DI void step_map(unsigned t, unsigned& j, unsigned& kc) {
    if (t < 8) { j = 0; kc = t; }
    else { const unsigned u = t - 8; kc = (u * 147u) >> 10; j = 1 + (u - 7 * kc); }
}

DI unsigned stage_map(unsigned s, unsigned q) {
    const unsigned o = (q + 2u) & 3u, v = (s - o) & 3u, s0 = s - v;
    if (s < o || s0 + 3u >= NSTEP || (s0 < 8u && s0 + 2u >= 8u)) return s;
    return v == 0 ? s + 2u : (v == 3 ? s : s - 1u);
}

DI unsigned pack_relu(float a, float b) {
    f32x2 v = {__builtin_elementwise_maximum(a, 0.f), __builtin_elementwise_maximum(b, 0.f)};
    bf16x2 r = __builtin_convertvector(v, bf16x2);
    return __builtin_bit_cast(unsigned, r);
}

__global__ __launch_bounds__(512, 2) __attribute__((amdgpu_num_vgpr(232))) void a2a_gemm(Params p) {
    extern __shared__ __attribute__((aligned(16))) char smem[];
    const unsigned ldsb = (unsigned)(size_t)(LDS char*)smem;
    const unsigned tid = threadIdx.x, lane = tid & 63;
    const unsigned wid = rfl(tid >> 6);
    const unsigned b = blockIdx.x;
    const unsigned mq = (b >> 3) & 3;
    const unsigned colb = (b & 7) * 4096u + (b >> 5) * 512u;
    const unsigned rank = (unsigned)p.rank & (NR - 1);
    const unsigned pi = (b >> 3) - (b >> 6), pq = (pi * 37u) >> 8;
    const bool pusher = ((b >> 3) & 7u) != 7u;
    const unsigned dst = (rank + 1u + (pi - pq * 7u)) & (NR - 1);
    char* const myws = p.myws;
    unsigned* const callp = (unsigned*)(myws + WS_CALL + b * 128);
    unsigned* const tmo = (unsigned*)(myws + WS_TMO);
#define GET_CALL() (rfl(*(const volatile unsigned __attribute__((address_space(4)))*)(size_t)callp) + 1u)
    ST_DECL;

    if (wid >= 4) {
        const unsigned lw = wid - 4;
        const unsigned tstart = (unsigned)__builtin_amdgcn_s_memrealtime();
        const __amdgpu_buffer_rsrc_t srdW = __builtin_amdgcn_make_buffer_rsrc((void*)p.w, (short)0, (int)W_BYTES, 0x00020000);
        const __amdgpu_buffer_rsrc_t srdX = __builtin_amdgcn_make_buffer_rsrc((void*)p.x, (short)0, (int)X_BYTES, 0x00020000);
        const __amdgpu_buffer_rsrc_t srdR = __builtin_amdgcn_make_buffer_rsrc((void*)myws, (short)0, (int)WS_CNT, 0x00020000);
        unsigned vW[4], vW2[4], vL[4];
        const unsigned vR = lane * 16u;
#pragma unroll
        for (int j = 0; j < 4; ++j) {
            vW[j] = (lane >> 5) * WROW + ((((lane >> 3) & 3) ^ (lane >> 5)) * 128u) + (((lane & 7) ^ ((j & 1) << 1)) * 16u) + j * (2u * WROW - 1024u);
            vW2[j] = vW[j] ^ 64u;
            vL[j] = (lane >> 3) * 1024u + (((lane & 7) ^ (4 * (j & 1) + (lane >> 4))) * 16u) + j * (8192u - 1024u);
        }
        const unsigned sW = colb + lw * 16u * WROW;
        constexpr unsigned WQ2 = 8u * WROW;
        const unsigned lW = ldsb + lw * 8192u;
        const unsigned sL = (rank * 512u + mq * 128u + lw * 32u) * 1024u;
        const unsigned lA = ldsb + A_BASE + lw * 4096u;
        unsigned tW, tA;
        { unsigned j, kc; step_map(stage_map(lane, mq), j, kc);
          const unsigned src = (rank + j) & (NR - 1);
          tW = sW + (src * KSH + kc * 64u) * WROW;
          tA = j == 0 ? sL + 128u * kc : mq * 16384u + lw * 4096u + src * WS_SRC + 65536u * kc; }
#define RL(t, s) __builtin_amdgcn_readlane(t, s)
#define ISSUE_W(mw, ow) do { DMA4("", mw, vW, srdW, ow); DMA4("", (mw) + 4096u, vW2, srdW, (ow) + WQ2); } while (0)
        __amdgpu_buffer_rsrc_t srdA = srdX;
        unsigned vA[4] = {vL[0], vL[1], vL[2], vL[3]};
#define ISSUE(mw, ow, ma, oa) do { ISSUE_W(mw, ow); DMA4("sc0 sc1", ma, vA, srdA, oa); } while (0)
        { const unsigned w0 = RL(tW, 0), a0 = RL(tA, 0), w1 = RL(tW, 1), a1 = RL(tA, 1);
          asm volatile("s_nop 4" ::: "memory");
          ISSUE(lW, w0, lA, a0); ISSUE(lW + B_SLOT, w1, lA + A_SLOT, a1); }
        unsigned ow = RL(tW, 2), oa = RL(tA, 2);
        const unsigned call = GET_CALL();
        WAITV(12);
        BAR();
        ISSUE(lW + 2 * B_SLOT, ow, lA + 2 * A_SLOT, oa);
        ow = RL(tW, 3); oa = RL(tA, 3);
        if (lw == 3) {
            const unsigned vp = (rank * 512u + mq * 128u + (b >> 5) * 16u + (lane & 15)) * 1024u + (4u + (lane >> 4)) * 128u;
            asm volatile("buffer_load_dword v220, %0, %1, 0 offen" :: "v"(vp), "s"(srdX) : "memory", "v220");
        }
        if (lane >= 8) tA += (call & 1u) * WS_PAR;
        const unsigned target = call * CNT_PER_CALL;
        unsigned long long ready = 0xffull;
        const unsigned* cptr;
        { unsigned j, kc; step_map(stage_map(8u + (lane < 56 ? lane : 55u), mq), j, kc);
          cptr = (const unsigned*)(myws + WS_CNT + (((rank + j) & (NR - 1)) * 4u + mq) * 128u + kc * 4u); }
        unsigned ksamp = 3;
        unsigned long long ev = (1ull << 61) | (1ull << 5);
        if (lw == 0 && ~ready != 0) ev |= 3ull << 3;
        unsigned mw = lW, ma = lA, k = 0;
#define ST_LINE()
#define STEP() do { ISSUE(mw, ow, ma, oa); ow = RL(tW, k + 4); oa = RL(tA, k + 4); ST_LINE(); \
            mw = (mw == lW + 2 * B_SLOT) ? lW : mw + B_SLOT; ma = (ma == lA + 2 * A_SLOT) ? lA : ma + A_SLOT; ++k; \
            asm volatile("" : "+s"(ow), "+s"(oa), "+s"(mw), "+s"(ma), "+s"(k)); } while (0)
        asm volatile("" : "+s"(ow), "+s"(oa), "+s"(mw), "+s"(ma), "+s"(k));
        WAITV(12);
#define PACE_PH 0
#define PACE_PM 0
        const unsigned tp0 = tstart + PACE_T0 + ((PACE_PM == 0 ? (b >> 5) : PACE_PM == 1 ? (b & 7u) : ((b >> 5) + (b & 7u)) & 7u) * PACE_PH >> 3) - PACE_PH / 2;
        BAR();
        for (;;) {
            const unsigned kend = rfl(k + (unsigned)__builtin_ctzll(ev >> k));
            if (lw == 3) while (k != kend) {
                STEP(); WAITV(12);
                { const unsigned due = tp0 + k * PACE - (k > PACE_N ? (k - PACE_N) * (PACE - PACE2) : 0u); while ((int)((unsigned)__builtin_amdgcn_s_memrealtime() - due) < 0) __builtin_amdgcn_s_sleep(1); }
                BAR();
            }
            while (k != kend) { STEP(); WAITV(12); BAR(); }
            if (k == NSTEP - 3) break;
            ev &= ~(1ull << k);
            if (k == 5) { srdA = srdR; vA[0] = vA[1] = vA[2] = vA[3] = vR; }
            if (lw == 0 && k == ksamp) {
                asm volatile("global_load_dword v221, %0, off sc0 sc1" :: "v"(cptr) : "memory", "v221");
                STEP(); WAITV(13);
            } else {
                STEP(); WAITV(12);
                if (lw == 0 && k == ksamp + 2) {
                    unsigned pv; asm volatile("v_mov_b32 %0, v221" : "=v"(pv) :: "memory");
                    ready |= __ballot(pv >= target) << 8;
                    if (!((ready >> (k + 3)) & 1ull)) {
                        const unsigned long long t0 = __builtin_amdgcn_s_memrealtime();
                        for (;;) {
                            unsigned v, to;
                            asm volatile("global_load_dword %0, %2, off sc0 sc1\n\tglobal_load_dword %1, %3, off sc1\n\ts_waitcnt vmcnt(0)"
                                         : "=&v"(v), "=&v"(to) : "v"(cptr), "v"(tmo) : "memory");
                            ready |= __ballot(v >= target) << 8;
                            if ((ready >> (k + 3)) & 1ull) break;
                            if (rfl(to) == call || __builtin_amdgcn_s_memrealtime() - t0 > DEADLINE_TICKS) {
                                asm volatile("global_store_dword %0, %1, off sc1\n\ts_waitcnt vmcnt(0)" :: "v"(tmo), "v"(call) : "memory");
                                ready = ~0ull;
                                break;
                            }
                            __builtin_amdgcn_s_sleep(2);
                        }
                    }
                    if (~ready != 0) {
                        const unsigned S = (unsigned)__builtin_ctzll(~ready);
                        const unsigned km = (S - 5u) - ((S - 5u + 1u - mq) & 3u);
                        ksamp = (int)km >= (int)k ? km : S - 5u;
                        ev |= ((ksamp >= k ? 1ull : 0ull) << ksamp) | (2ull << ksamp);
                    }
                }
            }
            BAR();
        }
        WAITV(0);
        BAR(); BAR(); BAR();
        if (lw == 0 && lane == 0) __hip_atomic_store(callp, call, __ATOMIC_RELAXED, __HIP_MEMORY_SCOPE_AGENT);
        ST_OUT(st_pollk);
    } else {
        const unsigned fr = lane & 15, fq = lane >> 4;
        const unsigned call = GET_CALL();
        unsigned long long evm = 0;
        char* dp = nullptr; char* cp = nullptr;
        const unsigned kc0 = (b & 7u) % PUSH_G, prow = (pq & 3u) * 128u + ((b & 7u) / PUSH_G) * (128u / PUSH_E) + wid * (32u / PUSH_E);
        const int e0 = PUSH_GAP * (int)kc0 - 1;
        constexpr int EP = PUSH_GAP * PUSH_G;
        static_assert(PUSH_GAP >= 2 && PUSH_GAP - 1 + PUSH_D <= 9 && PUSH_GAP <= 7 && PUSH_D < EP && EP - 1 > 4, "push schedule");
        const bool push = pusher;
        const char* const sp = (const char*)p.x + (size_t)(dst * 512u + prow) * 1024u + kc0 * 128u;
        const unsigned vs0 = ((lane >> 3) * 1024u + (((lane & 7) ^ (lane >> 4)) * 16u)) ^ ((prow & 8u) * 8u);
        const unsigned vs1 = PUSH_P > 1 ? (vs0 ^ 64u) + 8192u : vs0 + PUSH_G * 128u;
        constexpr unsigned SO2 = PUSH_P == 4 ? 16384u : PUSH_P == 2 ? PUSH_G * 128u : 2 * PUSH_G * 128u;
#define PUSH_LOAD() asm volatile("global_load_dwordx4 v[232:235], %0, %2 nt\n\tglobal_load_dwordx4 v[236:239], %1, %2 nt\n\t" \
                                 "global_load_dwordx4 v[240:243], %0, %3 nt\n\tglobal_load_dwordx4 v[244:247], %1, %3 nt" \
            :: "v"(vs0), "v"(vs1), "s"(sp), "s"(sp + SO2) : "memory", "v232", "v233", "v234", "v235", "v236", "v237", "v238", "v239", \
               "v240", "v241", "v242", "v243", "v244", "v245", "v246", "v247")
#define ST1(r, ptr, off) asm volatile("global_store_dwordx4 v248, " r ", %0 offset:" #off " sc0 sc1" :: "s"(ptr) : "memory")
        auto push_store = [&](int i) {
            char* const q = dp + i * PUSH_G * 65536u;
            if (PUSH_E == 1) { ST1("v[232:235]", q, 0); ST1("v[236:239]", q, 1024); ST1("v[240:243]", q, 2048); ST1("v[244:247]", q, 3072); }
            else if (PUSH_E == 2) { if (i == 0) { ST1("v[232:235]", q, 0); ST1("v[236:239]", q, 1024); } else { ST1("v[240:243]", q, 0); ST1("v[244:247]", q, 1024); } }
            else { if (i == 0) ST1("v[232:235]", q, 0); else if (i == 1) ST1("v[236:239]", q, 0); else if (i == 2) ST1("v[240:243]", q, 0); else ST1("v[244:247]", q, 0); }
        };
        if (push) {
            char* const pws = p.ws[dst];
            cp = pws + WS_CNT + (rank * 4u + (pq & 3u)) * 128u + kc0 * 4u;
            dp = pws + (call & 1u) * WS_PAR + rank * WS_SRC + kc0 * 65536u + prow * 128u;
            asm volatile("v_mov_b32 v248, %0\n\tv_mov_b32 v249, 0\n\tv_mov_b32 v250, 1" :: "v"(lane * 16u) : "v248", "v249", "v250");
#pragma unroll
            for (int i = 0; i < PUSH_E; ++i) if (e0 + i * EP >= 0) evm |= (1ull << (e0 + i * EP)) | (1ull << (e0 + i * EP + PUSH_D));
            if (kc0 == 0) {
                PUSH_LOAD();
                asm volatile("s_mov_b64 exec, 1\n\tglobal_atomic_add v249, v249, %0 sc1\n\tglobal_atomic_add v249, v249, %1 offset:64 sc1\n\ts_mov_b64 exec, -1\n\t"
                             "s_waitcnt vmcnt(%2)" :: "s"(dp), "s"(cp), "i"(2 + 4 - PUSH_P) : "memory");
                push_store(0);
                evm |= 0x1eu;
            }
        }
        auto push_event = [&](int k) {
            if (!((evm >> k) & 1ull)) return;
            const int d = k - e0, i = d / EP;
            if (d == i * EP) { WAITV(0); push_store(i); }
            else if (k > 4 || kc0 != 0 || k == 4 || __builtin_amdgcn_s_getreg((3 << 11) | 7) == 0) {
                asm volatile("s_waitcnt vmcnt(0)\n\ts_mov_b64 exec, 1\n\tglobal_atomic_add v249, v250, %0 sc1\n\ts_mov_b64 exec, -1" :: "s"(cp + i * PUSH_G * 4u) : "memory");
                evm &= ~0ull << (k < 4 ? 5 : k + 1);
            }
        };
        unsigned a0 = ldsb + A_BASE + 128u * fr + 16u * (fq ^ (fr >> 1));
        unsigned a1 = ldsb + A_BASE + 128u * fr + 16u * ((fq + 4) ^ (fr >> 1));
        const unsigned q = (lane >> 2) & 3, pp = lane & 3;
        const unsigned fB = (q & 2) | ((fq & 1) << 2);
        unsigned bb[4], bh[4];
#pragma unroll
        for (int c = 0; c < 4; ++c) {
            bb[c] = ldsb + 512u * (8 * fq + q) + 128u * (wid ^ (q & 1)) + 16u * ((2 * c + (pp >> 1)) ^ fB) + 8u * (pp & 1);
            bh[c] = bb[c] + 2 * B_SLOT;
        }
        f32x4 acc[8][4];
        bf16x8 X0[8], X1[8];

        s16x4 Wl0[4], Wh0[4], Wl1[4], Wh1[4];
#define WFRAG(S, c) __builtin_bit_cast(bf16x8, __builtin_shufflevector(Wl##S[c], Wh##S[c], 0, 1, 2, 3, 4, 5, 6, 7))
#define MF(S, i) do { const int m_ = (i) >> 2, c_ = (m_ & 1) ? 3 - ((i) & 3) : ((i) & 3); \
            asm volatile("v_mfma_f32_16x16x32_bf16 %0, %1, %2, %0" : "+v"(acc[m_][c_]) : "v"(WFRAG(S, c_)), "v"(X##S[m_]) : "memory"); } while (0)
#define MF0(S, i) do { const int m_ = (i) >> 2, c_ = (m_ & 1) ? 3 - ((i) & 3) : ((i) & 3); \
            asm volatile("v_mfma_f32_16x16x32_bf16 %0, %1, %2, 0" : "=&v"(acc[m_][c_]) : "v"(WFRAG(S, c_)), "v"(X##S[m_]) : "memory"); } while (0)
#define RD(S, r, j) do { if ((j) < 8) { const unsigned o_ = ((r) == 2 ? bh[((j) >> 1) & 3] : bb[((j) >> 1) & 3] + (r) * B_SLOT) + ((S) * 4096u + ((j) & 1) * 512u) * (WLAY == 1 ? 4u : 1u); \
                if ((j) & 1) Wh##S[((j) >> 1) & 3] = __builtin_amdgcn_ds_read_tr16_b64_v4i16((LDS s16x4*)o_); \
                else         Wl##S[((j) >> 1) & 3] = __builtin_amdgcn_ds_read_tr16_b64_v4i16((LDS s16x4*)o_); } \
            else X##S[(j) & 7] = *(const LDS bf16x8*)(((S) ? a1 : a0) + (r) * A_SLOT + 2048u * ((j) & 7)); } while (0)
#define RUN28(MFX, SM, i0, SR, r) do { _Pragma("unroll") for (int g = 0; g < 4; ++g) { \
            MFX(SM, (i0) + 7 * g); RD(SR, r, 4 * g); MFX(SM, (i0) + 7 * g + 1); MFX(SM, (i0) + 7 * g + 2); RD(SR, r, 4 * g + 1); \
            MFX(SM, (i0) + 7 * g + 3); MFX(SM, (i0) + 7 * g + 4); RD(SR, r, 4 * g + 2); MFX(SM, (i0) + 7 * g + 5); MFX(SM, (i0) + 7 * g + 6); RD(SR, r, 4 * g + 3); } } while (0)
#define RUN4(MFX, SM, i0) do { MFX(SM, i0); MFX(SM, (i0) + 1); MFX(SM, (i0) + 2); MFX(SM, (i0) + 3); } while (0)
#define TO_BARRIER() do { __builtin_amdgcn_s_waitcnt(0xC07F); ST_BEG(); BAR(); ST_END(st_bar); } while (0)
#define EVPOS 2
#define BODY(r, i) do { RUN28(MF, 1, 4, 0, r); RUN28(MF, 0, 0, 1, r); RUN4(MF, 0, 28); \
            _Pragma("unroll") for (int j = 0; j < EVPOS; ++j) MF(1, j); \
            if (__builtin_expect((em >> (i)) & 1u, 0)) push_event(kb + (i)); \
            _Pragma("unroll") for (int j = EVPOS; j < 4; ++j) MF(1, j); TO_BARRIER(); } while (0)

        asm volatile("" : "+v"(a0), "+v"(a1), "+v"(bb[0]), "+v"(bb[1]), "+v"(bb[2]), "+v"(bb[3]), "+v"(bh[0]), "+v"(bh[1]), "+v"(bh[2]), "+v"(bh[3]));
        BAR();
        if (push && kc0 != 0) PUSH_LOAD();
        ST_FIRST();
#pragma unroll
        for (int j = 0; j < 16; ++j) RD(0, 0, j);
        RUN28(MF0, 0, 0, 1, 0); RUN4(MF0, 0, 28); RUN4(MF, 1, 0);
        TO_BARRIER();
        for (int kb = 1; kb < NSTEP - 3; kb += 3) {
            unsigned em = (unsigned)(evm >> kb);
            asm volatile("" : "+s"(em));
            BODY(1, 0); BODY(2, 1); BODY(0, 2);
        }
        { const unsigned em = 0; const int kb = NSTEP - 3; BODY(1, 0); }
        const unsigned tv = __hip_atomic_load(tmo, __ATOMIC_RELAXED, __HIP_MEMORY_SCOPE_AGENT);
        char* const orow = (char*)p.out + (size_t)(mq * 128 + fr) * WROW + colb + wid * 128u + (16 * (fq & 1) + 8 * (fq >> 1)) * 2;
        unsigned pk[4];
#define EPI(u, t) do { const int m_ = (u) >> 1, h_ = (u) & 1; \
            if ((t) == 0) asm volatile("" : "+v"(acc[m_][2 * h_]), "+v"(acc[m_][2 * h_ + 1])); \
            if ((t) < 4) { const f32x4 a_ = acc[m_][2 * h_ + ((t) >> 1 & 1)]; pk[(t) & 3] = pack_relu(a_[2 * ((t) & 1)], a_[2 * ((t) & 1) + 1]); asm volatile("" : "+v"(pk[(t) & 3])); } \
            else if ((t) == 4) { const auto s0 = __builtin_amdgcn_permlane16_swap(pk[0], pk[2], false, false); const auto s1 = __builtin_amdgcn_permlane16_swap(pk[1], pk[3], false, false); \
                pk[0] = s0[0]; pk[2] = s0[1]; pk[1] = s1[0]; pk[3] = s1[1]; asm volatile("" : "+v"(pk[0]), "+v"(pk[1]), "+v"(pk[2]), "+v"(pk[3])); } \
            else { const u32x4 o_ = {pk[0], pk[1], pk[2], pk[3]}; \
                if (h_) asm volatile("global_store_dwordx4 %0, %1, off offset:64 sc0 sc1\n\ts_nop 1" :: "v"(orow + (size_t)m_ * 16 * WROW), "v"(o_) : "memory"); \
                else    asm volatile("global_store_dwordx4 %0, %1, off sc0 sc1\n\ts_nop 1" :: "v"(orow + (size_t)m_ * 16 * WROW), "v"(o_) : "memory"); } } while (0)
#define RDH(S, r, H, i) RD(S, r, (i) < 2 ? (i) : (i) == 2 ? 8 + 4 * (H) : (i) < 9 ? (i) - 1 : (i) + 4 * (H))
#define PASS(SM, H, SR, r, HR, g0) do { _Pragma("unroll") for (int i = 0; i < 16; ++i) { MF(SM, 16 * (H) + i); \
            if ((r) >= 0 && i < 12) RDH(SR, (r) < 0 ? 0 : (r), HR, i); \
            if ((g0) >= 0 && (((g0) + i) & 3) != 3) { const int s_ = ((g0) + i) - (((g0) + i) >> 2); EPI(s_ / 6, s_ % 6); } } } while (0)
#pragma unroll
        for (int i = 4; i < 32; ++i) { MF(1, i); if (i >= 8 && !(i & 1)) RDH(0, 2, 0, (i - 8) >> 1); }
        PASS(0, 0, 1, 2, 0, -1);
        BAR();
        PASS(1, 0, 0, 0, 0, -1); PASS(0, 0, 1, 0, 0, -1); PASS(1, 0, 0, 2, 1, -1);
        unsigned tvs = rfl(tv);
        asm volatile("" : "+s"(tvs));
        BAR();
        PASS(0, 1, 1, 2, 1, 0); PASS(1, 1, 0, 0, 1, 16); PASS(0, 1, 1, 0, 1, 32); PASS(1, 1, 0, -1, 1, 48);
        asm volatile("s_nop 7\n\ts_nop 1" ::: "memory");
#pragma unroll
        for (int s_ = 48; s_ < 96; ++s_) EPI(s_ / 6, s_ % 6);
        if (tvs == call) {
            WAITV(0);
            const u32x4 o_ = {0x7fc07fc0u, 0x7fc07fc0u, 0x7fc07fc0u, 0x7fc07fc0u};
#pragma unroll
            for (int m = 0; m < 8; ++m)
                asm volatile("global_store_dwordx4 %0, %1, off sc0 sc1\n\tglobal_store_dwordx4 %0, %1, off offset:64 sc0 sc1\n\ts_nop 1"
                             :: "v"(orow + (size_t)m * 16 * WROW), "v"(o_) : "memory");
        }
        BAR();
    }
}

}

extern "C" void kernel_launch(int rank, int nranks, ncclComm_t comm,
                              hipStream_t stream,
                              void* const* d_in, const int* in_sizes, int n_in,
                              void* d_out, int out_size,
                              const int* in_dtypes, int out_dtype,
                              void* const* d_out_all,
                              void* const* d_ws_all, size_t ws_bytes) {
    using namespace a2ag;
    if (nranks != NR || (unsigned)rank >= (unsigned)NR || n_in < 2 || ws_bytes < WS_END ||
        in_sizes[0] != 4096 * KSH || in_sizes[1] != KTOT * NTOT || out_size != MLOC * NTOT) {
        fprintf(stderr, "kernel_launch: unexpected shapes\n");
        return;
    }
    HIP_CHECK(hipFuncSetAttribute((const void*)a2a_gemm, hipFuncAttributeMaxDynamicSharedMemorySize, (int)LDS_BYTES));
    Params p;
    p.x = d_in[0]; p.w = d_in[1]; p.out = d_out; p.myws = (char*)d_ws_all[rank];
    for (int i = 0; i < NR; ++i) p.ws[i] = (char*)d_ws_all[i];
    p.rank = rank; p.pad = 0;
    hipLaunchKernelGGL(a2a_gemm, dim3(NWG), dim3(512), LDS_BYTES, stream, p);
    HIP_CHECK(hipGetLastError());
}
```

```cpp
#include <hip/hip_runtime.h>
#include <rccl/rccl.h>
#include <cstdio>

#define HIP_CHECK(expr) do { hipError_t _e = (expr); if (_e != hipSuccess) { \
    fprintf(stderr, "HIP error %s at %s:%d\n", hipGetErrorString(_e), __FILE__, __LINE__); \
    return; } } while (0)

namespace a2ag {

#define LDS __attribute__((address_space(3)))
#define DI __device__ __forceinline__

constexpr int NR = 8;
constexpr int MLOC = 512;
constexpr int KSH = 512;
constexpr int KTOT = 4096, NTOT = 16384;
constexpr int NSTEP = KTOT / 64;
constexpr int NWG = 256;
constexpr unsigned B_SLOT = 32768, A_SLOT = 16384;
constexpr unsigned A_BASE = 3 * B_SLOT;
constexpr unsigned LDS_BYTES = A_BASE + 3 * A_SLOT;
constexpr unsigned X_BYTES = 4096u * KSH * 2u;
constexpr unsigned W_BYTES = (unsigned)KTOT * NTOT * 2u;
constexpr unsigned WROW = NTOT * 2u;

constexpr unsigned WS_SRC = 512u * 1024u;
constexpr unsigned WS_PAR = NR * WS_SRC;
constexpr unsigned WS_CNT = 2 * WS_PAR;
constexpr unsigned WS_CALL = WS_CNT + NR * 4 * 128;
constexpr unsigned WS_TMO = WS_CALL + NWG * 128;
constexpr unsigned WS_END = WS_TMO + 128;
constexpr unsigned CNT_PER_CALL = 16;
constexpr unsigned long long DEADLINE_TICKS = 50000000ull;

constexpr int PUSH_GAP = 3;
constexpr int PUSH_D = 4;
constexpr unsigned PACE_T0 = 270, PACE_A = 73, PACE_NA = 10, PACE_B = 86, PACE_NB = 38, PACE_C = 68;

struct Params {
    const void* x; const void* w; void* out; char* myws;
    char* ws[NR];
    int rank; int pad;
};

#define BAR() do { asm volatile("" ::: "memory"); __builtin_amdgcn_s_barrier(); asm volatile("" ::: "memory"); } while (0)
#define WAITV(n) asm volatile("s_waitcnt vmcnt(" #n ")" ::: "memory")

DI unsigned rfl(unsigned v) { return __builtin_amdgcn_readfirstlane(v); }

#define DMA4(suffix, lds, v, srd, soff) asm volatile("s_mov_b32 m0, %0\n\ts_nop 0\n\t" \
        "buffer_load_dwordx4 %1, %5, %6 offen " suffix " lds\n\t" \
        "buffer_load_dwordx4 %2, %5, %6 offen offset:1024 " suffix " lds\n\t" \
        "buffer_load_dwordx4 %3, %5, %6 offen offset:2048 " suffix " lds\n\t" \
        "buffer_load_dwordx4 %4, %5, %6 offen offset:3072 " suffix " lds" \
        :: "s"(lds), "v"((v)[0]), "v"((v)[1]), "v"((v)[2]), "v"((v)[3]), "s"(srd), "s"(soff) : "memory")

DI void step_map(unsigned t, unsigned& j, unsigned& kc) {
    if (t < 8) { j = 0; kc = t; }
    else { const unsigned u = t - 8; kc = (u * 147u) >> 10; j = 1 + (u - 7 * kc); }
}

DI unsigned stage_map(unsigned s, unsigned q) {
    const unsigned o = (q + 2u) & 3u, v = (s - o) & 3u, s0 = s - v;
    if (s < o || s0 + 3u >= NSTEP || (s0 < 8u && s0 + 2u >= 8u)) return s;
    return v == 0 ? s + 2u : (v == 3 ? s : s - 1u);
}

__global__ __launch_bounds__(512, 2) void a2a_gemm(Params p) {
    extern __shared__ __attribute__((aligned(16))) char smem[];
    const unsigned ldsb = (unsigned)(size_t)(LDS char*)smem;
    const unsigned wid = rfl(threadIdx.x >> 6);
#define GET_LANE(l) unsigned l; asm volatile("v_mbcnt_lo_u32_b32 %0, -1, 0\n\tv_mbcnt_hi_u32_b32 %0, -1, %0" : "=v"(l))
    const void* px = p.x; const void* pw = p.w; void* pout = p.out; char* myws = p.myws; int prank = p.rank;
    asm volatile("" : "+s"(px), "+s"(pw), "+s"(pout), "+s"(myws), "+s"(prank));
    const unsigned b = blockIdx.x;
    const unsigned mq = (b >> 3) & 3;
    const unsigned colb = (b & 7) * 4096u + (b >> 5) * 512u;
    const unsigned rank = (unsigned)prank & (NR - 1);
    const unsigned pi = (b >> 3) - (b >> 6), pq = (pi * 37u) >> 8;
    const bool pusher = ((b >> 3) & 7u) != 7u;
    const unsigned dst = (rank + 1u + (pi - pq * 7u)) & (NR - 1);
    unsigned* const callp = (unsigned*)(myws + WS_CALL + b * 128);
    unsigned* const tmo = (unsigned*)(myws + WS_TMO);
#define GET_CALL() (rfl(*(const volatile unsigned __attribute__((address_space(4)))*)(size_t)callp) + 1u)

    if (wid >= 4) {
        const unsigned lw = wid - 4;
        GET_LANE(lane);
        const unsigned tp0 = (unsigned)__builtin_amdgcn_s_memrealtime() + PACE_T0;
        const __amdgpu_buffer_rsrc_t srdW = __builtin_amdgcn_make_buffer_rsrc((void*)pw, (short)0, (int)W_BYTES, 0x00020000);
        const __amdgpu_buffer_rsrc_t srdX = __builtin_amdgcn_make_buffer_rsrc((void*)px, (short)0, (int)X_BYTES, 0x00020000);
        const __amdgpu_buffer_rsrc_t srdR = __builtin_amdgcn_make_buffer_rsrc((void*)myws, (short)0, (int)WS_CNT, 0x00020000);
        unsigned vW[4], vW2[4], vL[4];
        const unsigned vR = lane * 16u;
#pragma unroll
        for (int j = 0; j < 4; ++j) {
            vW[j] = (lane >> 5) * WROW + ((((lane >> 3) & 3) ^ (lane >> 5)) * 128u) + (((lane & 7) ^ ((j & 1) << 1)) * 16u) + j * (2u * WROW - 1024u);
            vW2[j] = vW[j] ^ 64u;
            vL[j] = (lane >> 3) * 1024u + (((lane & 7) ^ (4 * (j & 1) + (lane >> 4))) * 16u) + j * (8192u - 1024u);
        }
        const unsigned sW = colb + lw * 16u * WROW;
        constexpr unsigned WQ2 = 8u * WROW;
        const unsigned lW = ldsb + lw * 8192u;
        const unsigned sL = (rank * 512u + mq * 128u + lw * 32u) * 1024u;
        const unsigned lA = ldsb + A_BASE + lw * 4096u;
#define RL(t, s) __builtin_amdgcn_readlane(t, s)
#define ISSUE_W(mw, ow) do { DMA4("", mw, vW, srdW, ow); DMA4("", (mw) + 4096u, vW2, srdW, (ow) + WQ2); } while (0)
        __amdgpu_buffer_rsrc_t srdA = srdX;
        unsigned vA[4] = {vL[0], vL[1], vL[2], vL[3]};
#define ISSUE(mw, ow, ma, oa) do { ISSUE_W(mw, ow); DMA4("sc0 sc1", ma, vA, srdA, oa); } while (0)
        { const unsigned t0 = stage_map(0, mq), t1 = stage_map(1, mq), w0 = sW + rank * KSH * WROW;
          ISSUE(lW, w0 + t0 * 64u * WROW, lA, sL + 128u * t0); ISSUE(lW + B_SLOT, w0 + t1 * 64u * WROW, lA + A_SLOT, sL + 128u * t1); }
        unsigned tW, tA;
        { unsigned j, kc; step_map(stage_map(lane, mq), j, kc);
          const unsigned src = (rank + j) & (NR - 1);
          tW = sW + (src * KSH + kc * 64u) * WROW;
          tA = j == 0 ? sL + 128u * kc : mq * 16384u + lw * 4096u + src * WS_SRC + 65536u * kc; }
        unsigned ow = RL(tW, 2), oa = RL(tA, 2);
        const unsigned call = GET_CALL();
        WAITV(12);
        BAR();
        ISSUE(lW + 2 * B_SLOT, ow, lA + 2 * A_SLOT, oa);
        ow = RL(tW, 3); oa = RL(tA, 3);
        if (lw == 3) {
            const unsigned vp = (rank * 512u + mq * 128u + (b >> 5) * 16u + (lane & 15)) * 1024u + (4u + (lane >> 4)) * 128u;
            asm volatile("buffer_load_dword v220, %0, %1, 0 offen" :: "v"(vp), "s"(srdX) : "memory", "v220");
        }
        if (lane >= 8) tA += (call & 1u) * WS_PAR;
        const unsigned target = call * CNT_PER_CALL;
        unsigned long long ready = 0xffull;
        const unsigned* cptr;
        { unsigned j, kc; step_map(stage_map(8u + (lane < 56 ? lane : 55u), mq), j, kc);
          cptr = (const unsigned*)(myws + WS_CNT + (((rank + j) & (NR - 1)) * 4u + mq) * 128u + kc * 4u); }
        unsigned ksamp = 3;
        unsigned long long ev = (1ull << 61) | (1ull << 5);
        if (lw == 0) ev |= 3ull << 3;
        unsigned mw = lW, ma = lA, k = 0;
#define STEP() do { ISSUE(mw, ow, ma, oa); ow = RL(tW, k + 4); oa = RL(tA, k + 4); \
            mw = (mw == lW + 2 * B_SLOT) ? lW : mw + B_SLOT; ma = (ma == lA + 2 * A_SLOT) ? lA : ma + A_SLOT; ++k; \
            asm volatile("" : "+s"(ow), "+s"(oa), "+s"(mw), "+s"(ma), "+s"(k)); } while (0)
        asm volatile("" : "+s"(ow), "+s"(oa), "+s"(mw), "+s"(ma), "+s"(k));
        WAITV(12);
        BAR();
        for (;;) {
            const unsigned kend = rfl(k + (unsigned)__builtin_ctzll(ev >> k));
            if (lw == 3) while (k != kend) {
                STEP(); WAITV(12);
                const unsigned due = tp0 + k * PACE_A + (k > PACE_NA ? (k - PACE_NA) * (PACE_B - PACE_A) : 0u) - (k > PACE_NB ? (k - PACE_NB) * (PACE_B - PACE_C) : 0u);
                while ((int)((unsigned)__builtin_amdgcn_s_memrealtime() - due) < 0) __builtin_amdgcn_s_sleep(1);
                BAR();
            }
            while (k != kend) { STEP(); WAITV(12); BAR(); }
            if (k == NSTEP - 3) break;
            ev &= ~(1ull << k);
            if (k == 5) { srdA = srdR; vA[0] = vA[1] = vA[2] = vA[3] = vR; }
            if (lw == 0 && k == ksamp) {
                asm volatile("global_load_dword v221, %0, off sc0 sc1" :: "v"(cptr) : "memory", "v221");
                STEP(); WAITV(13);
            } else {
                STEP(); WAITV(12);
                if (lw == 0 && k == ksamp + 2) {
                    unsigned pv; asm volatile("v_mov_b32 %0, v221" : "=v"(pv) :: "memory");
                    ready |= __ballot(pv >= target) << 8;
                    if (!((ready >> (k + 3)) & 1ull)) {
                        const unsigned long long t0 = __builtin_amdgcn_s_memrealtime();
                        for (;;) {
                            unsigned v, to;
                            asm volatile("global_load_dword %0, %2, off sc0 sc1\n\tglobal_load_dword %1, %3, off sc1\n\ts_waitcnt vmcnt(0)"
                                         : "=&v"(v), "=&v"(to) : "v"(cptr), "v"(tmo) : "memory");
                            ready |= __ballot(v >= target) << 8;
                            if ((ready >> (k + 3)) & 1ull) break;
                            if (rfl(to) == call || __builtin_amdgcn_s_memrealtime() - t0 > DEADLINE_TICKS) {
                                asm volatile("global_store_dword %0, %1, off sc1\n\ts_waitcnt vmcnt(0)" :: "v"(tmo), "v"(call) : "memory");
                                ready = ~0ull;
                                break;
                            }
                            __builtin_amdgcn_s_sleep(2);
                        }
                    }
                    if (~ready != 0) {
                        const unsigned S = (unsigned)__builtin_ctzll(~ready);
                        const unsigned km = (S - 5u) - ((S - 5u + 1u - mq) & 3u);
                        ksamp = (int)km >= (int)k ? km : S - 5u;
                        ev |= ((ksamp >= k ? 1ull : 0ull) << ksamp) | (2ull << ksamp);
                    }
                }
            }
            BAR();
        }
        WAITV(0);
        BAR(); BAR(); BAR();
        if (lw == 0) {
            unsigned t0, t1;
            asm volatile("v_mov_b32 %0, 0\n\tv_mov_b32 %1, %2\n\ts_mov_b64 exec, 1\n\tglobal_store_dword %0, %1, %3 sc1\n\ts_mov_b64 exec, -1"
                         : "=&v"(t0), "=&v"(t1) : "s"(call), "s"(callp) : "memory");
        }
    } else {
        constexpr int XA = 128, XB = 160, WF = 192;
        GET_LANE(lane);
        const unsigned fr = lane & 15, fq = lane >> 4, q = (lane >> 2) & 3, pp = lane & 3;
        const unsigned kc0 = b & 1u, prow = (pq & 3u) * 128u + ((b & 7u) >> 1) * 32u + wid * 8u;
        {
            const unsigned a0 = ldsb + A_BASE + 128u * fr + 16u * (fq ^ (fr >> 1));
            const unsigned b0 = ldsb + 512u * (8 * fq + q) + 128u * (wid ^ (q & 1)) + 16u * ((pp >> 1) ^ ((q & 2) | ((fq & 1) << 2))) + 8u * (pp & 1);
            const unsigned vo = fr * WROW + 32u * (fq & 1) + 16u * (fq >> 1);
            const unsigned vs = ((lane >> 3) * 1024u + (((lane & 7) ^ (lane >> 4)) * 16u)) ^ ((prow & 8u) * 8u);
            asm volatile("v_mov_b32 v224, %0\n\tv_mov_b32 v226, %1\n\tv_mov_b32 v234, %2\n\tv_mov_b32 v235, %3\n\tv_mov_b32 v238, %4\n\t"
                         "v_xor_b32 v225, 64, v224\n\tv_xor_b32 v227, 32, v226\n\tv_xor_b32 v228, 64, v226\n\tv_xor_b32 v229, 0x60, v226\n\t"
                         "v_add_u32 v230, 0x10000, v226\n\tv_add_u32 v231, 0x10000, v227\n\tv_add_u32 v232, 0x10000, v228\n\tv_add_u32 v233, 0x10000, v229\n\t"
                         "v_mov_b32 v236, 0\n\tv_mov_b32 v237, 1"
                         :: "v"(a0), "v"(b0), "v"(vo), "v"(lane * 16u), "v"(vs) : "v224", "v225", "v226", "v227", "v228", "v229", "v230", "v231",
                            "v232", "v233", "v234", "v235", "v236", "v237", "v238");
            asm volatile("" ::: "v0", "v1", "v2", "v3", "v4", "v5", "v6", "v7", "v8", "v9", "v10", "v11", "v12", "v13", "v14", "v15", "v16", "v17", "v18", "v19", "v20", "v21", "v22",
            "v23", "v24", "v25", "v26", "v27", "v28", "v29", "v30", "v31", "v32", "v33", "v34", "v35", "v36", "v37", "v38", "v39", "v40", "v41", "v42", "v43",
            "v44", "v45", "v46", "v47", "v48", "v49", "v50", "v51", "v52", "v53", "v54", "v55", "v56", "v57", "v58", "v59", "v60", "v61", "v62", "v63", "v64",
            "v65", "v66", "v67", "v68", "v69", "v70", "v71", "v72", "v73", "v74", "v75", "v76", "v77", "v78", "v79", "v80", "v81", "v82", "v83", "v84", "v85",
            "v86", "v87", "v88", "v89", "v90", "v91", "v92", "v93", "v94", "v95", "v96", "v97", "v98", "v99", "v100", "v101", "v102", "v103", "v104", "v105",
            "v106", "v107", "v108", "v109", "v110", "v111", "v112", "v113", "v114", "v115", "v116", "v117", "v118", "v119", "v120", "v121", "v122", "v123",
            "v124", "v125", "v126", "v127", "v128", "v129", "v130", "v131", "v132", "v133", "v134", "v135", "v136", "v137", "v138", "v139", "v140", "v141",
            "v142", "v143", "v144", "v145", "v146", "v147", "v148", "v149", "v150", "v151", "v152", "v153", "v154", "v155", "v156", "v157", "v158", "v159",
            "v160", "v161", "v162", "v163", "v164", "v165", "v166", "v167", "v168", "v169", "v170", "v171", "v172", "v173", "v174", "v175", "v176", "v177",
            "v178", "v179", "v180", "v181", "v182", "v183", "v184", "v185", "v186", "v187", "v188", "v189", "v190", "v191", "v192", "v193", "v194", "v195",
            "v196", "v197", "v198", "v199", "v200", "v201", "v202", "v203", "v204", "v205", "v206", "v207", "v208", "v209", "v210", "v211", "v212", "v213",
            "v214", "v215", "v216", "v217", "v218", "v219", "v220", "v221", "v222", "v223", "v224", "v225", "v226", "v227", "v228", "v229", "v230", "v231",
            "v232", "v233", "v234", "v235", "v236", "v237", "v238", "v239", "v240", "v241", "v242", "v243", "v244", "v245", "v246", "v247", "v248", "v249",
            "v250", "v251", "v252", "v253", "v254", "v255");
        }
        const unsigned call = GET_CALL();
        unsigned long long evm = 0;
        char* dp = nullptr; char* cp = nullptr;
        const int e0 = PUSH_GAP * (int)kc0 - 1;
        constexpr int EP = PUSH_GAP * 2;
        static_assert(PUSH_GAP >= 3 && PUSH_GAP - 1 + PUSH_D <= 9 && PUSH_GAP <= 7 && PUSH_D < EP, "push schedule");
        const char* const sp = (const char*)px + (size_t)(dst * 512u + prow) * 1024u + kc0 * 128u;
#define PUSH_LOAD() asm volatile("global_load_dwordx4 v[240:243], v238, %0 nt\n\tglobal_load_dwordx4 v[244:247], v238, %0 offset:256 nt\n\t" \
            "global_load_dwordx4 v[248:251], v238, %0 offset:512 nt\n\tglobal_load_dwordx4 v[252:255], v238, %0 offset:768 nt" :: "s"(sp) : "memory")
#define PUSH_STORE(i) asm volatile("global_store_dwordx4 v235, v[%c1:%c2], %0 sc0 sc1" :: "s"(dp + (i) * 131072u), "i"(240 + 4 * (i)), "i"(243 + 4 * (i)) : "memory")
        if (pusher) {
            char* const pws = p.ws[dst];
            cp = pws + WS_CNT + (rank * 4u + (pq & 3u)) * 128u + kc0 * 4u;
            dp = pws + (call & 1u) * WS_PAR + rank * WS_SRC + kc0 * 65536u + prow * 128u;
#pragma unroll
            for (int i = 0; i < 4; ++i) if (e0 + i * EP >= 0) evm |= (1ull << (e0 + i * EP)) | (1ull << (e0 + i * EP + PUSH_D));
            if (kc0 == 0) {
                PUSH_LOAD();
                asm volatile("s_mov_b64 exec, 1\n\tglobal_load_dword v239, v236, %0 sc0 sc1\n\tglobal_load_dword v239, v236, %1 sc0 sc1\n\ts_mov_b64 exec, -1\n\t"
                             "s_waitcnt vmcnt(5)" :: "s"(dp), "s"(cp) : "memory");
                PUSH_STORE(0);
                evm |= 0x1eu;
            }
        }
        auto push_event = [&](int k) {
            if (!((evm >> k) & 1ull)) return;
            const int d = k - e0;
            if (d == 0) { WAITV(0); PUSH_STORE(0); } else if (d == EP) { WAITV(0); PUSH_STORE(1); }
            else if (d == 2 * EP) { WAITV(0); PUSH_STORE(2); } else if (d == 3 * EP) { WAITV(0); PUSH_STORE(3); }
            else if (k >= 4 || __builtin_amdgcn_s_getreg((3 << 11) | 7) == 0) {
                const unsigned i = ((unsigned)d * (65536u / EP + 1u)) >> 16;
                asm volatile("s_waitcnt vmcnt(0)\n\ts_mov_b64 exec, 1\n\tglobal_atomic_add v236, v237, %0 sc1\n\ts_mov_b64 exec, -1" :: "s"(cp + i * 8u) : "memory");
                evm &= ~0ull << (k < 4 ? 5 : k + 1);
            }
        };
#define WAITL(n) asm volatile("s_waitcnt lgkmcnt(" #n ")" ::: "memory")
#define MF(i, H, XU, first) do { constexpr int c_ = (i) >> 3, m_ = (c_ & 1) ? 3 - ((i) >> 1 & 3) : ((i) >> 1 & 3), s_ = (((i) + 1) >> 1) & 1; \
            constexpr int d_ = 16 * (4 * (H) + m_) + 4 * c_, w_ = WF + 8 * c_ + 4 * s_, x_ = (XU) + 16 * s_ + 4 * m_; \
            if ((first) && !((i) & 1)) asm volatile("v_mfma_f32_16x16x32_bf16 v[%c0:%c1], v[%c2:%c3], v[%c4:%c5], 0" :: "i"(d_), "i"(d_ + 3), "i"(w_), "i"(w_ + 3), "i"(x_), "i"(x_ + 3) : "memory"); \
            else asm volatile("v_mfma_f32_16x16x32_bf16 v[%c0:%c1], v[%c2:%c3], v[%c4:%c5], v[%c0:%c1]" :: "i"(d_), "i"(d_ + 3), "i"(w_), "i"(w_ + 3), "i"(x_), "i"(x_ + 3) : "memory"); } while (0)
#define RW(r, c, t) asm volatile("ds_read_b64_tr_b16 v[%c0:%c1], v%c2 offset:%c3" :: "i"(WF + 8 * (c) + 2 * (t)), "i"(WF + 8 * (c) + 2 * (t) + 1), \
            "i"(((r) == 2 ? 230 : 226) + (c)), "i"(((r) & 1) * B_SLOT + ((t) >> 1) * 16384u + ((t) & 1) * 2048u) : "memory")
#define RX(XL, r, h, t) asm volatile("ds_read_b128 v[%c0:%c1], v%c2 offset:%c3" :: "i"((XL) + 16 * ((t) & 1) + 4 * ((t) >> 1)), "i"((XL) + 16 * ((t) & 1) + 4 * ((t) >> 1) + 3), \
            "i"(224 + ((t) & 1)), "i"((r) * A_SLOT + 2048u * (4 * (h) + ((t) >> 1))) : "memory")
        char* const obase = (char*)pout + (size_t)(mq * 128u) * WROW + colb + wid * 128u;
#define EPI(u, t) do { constexpr int a_ = 16 * ((u) >> 1) + 8 * ((u) & 1), t_ = (t); \
            if (t_ < 4) asm volatile("v_maximum3_f32 v%c0, v%c0, 0, 0\n\tv_maximum3_f32 v%c1, v%c1, 0, 0\n\tv_cvt_pk_bf16_f32 v%c2, v%c0, v%c1" \
                                     :: "i"(a_ + 2 * (t_ & 3)), "i"(a_ + 2 * (t_ & 3) + 1), "i"(a_ + (t_ & 3)) : "memory"); \
            else if (t_ == 4) asm volatile("s_nop 1\n\tv_permlane16_swap_b32 v%c0, v%c2\n\tv_permlane16_swap_b32 v%c1, v%c3" :: "i"(a_), "i"(a_ + 1), "i"(a_ + 2), "i"(a_ + 3) : "memory"); \
            else asm volatile("global_store_dwordx4 v234, v[%c1:%c2], %0 offset:%c3 sc0 sc1" :: "s"(obase + (size_t)((u) >> 1) * 16 * WROW), "i"(a_), "i"(a_ + 3), "i"(64 * ((u) & 1)) : "memory"); } while (0)
#define GAP(i, first, H, XU, w3, XL, rx, hx, wn, g0) do { \
            if ((w3) >= 0) { if ((i) == 0) WAITL(8); if ((i) == 8) { if ((rx) >= 0) WAITL(12); else WAITL(8); } \
                             if ((i) == 16) { if ((wn) >= 0) WAITL(15); else if ((rx) >= 0) WAITL(12); else WAITL(4); } \
                             if ((i) == 24) { if ((wn) >= 0) WAITL(8); else WAITL(0); } } \
            MF(i, H, XU, first); \
            if ((w3) >= 0 && (i) < 8 && !((i) & 1)) RW((w3) < 0 ? 0 : (w3), 3, ((i) >> 1) & 3); \
            if ((rx) >= 0 && (i) < 16 && ((i) & 1)) RX(XL, (rx) < 0 ? 0 : (rx), hx, ((i) >> 1) & 7); \
            if ((wn) >= 0 && (i) >= 8 && !((i) & 1)) RW((wn) < 0 ? 0 : (wn), (((i) + 24) >> 3) & 3, ((i) >> 1) & 3); \
            if ((g0) >= 0 && ((i) & 3) != 3) EPI((((g0) < 0 ? 0 : (g0)) + (i) - ((i) >> 2)) / 6, (((g0) < 0 ? 0 : (g0)) + (i) - ((i) >> 2)) % 6); } while (0)
#define GAP4(i, ...) GAP(i, __VA_ARGS__); GAP((i) + 1, __VA_ARGS__); GAP((i) + 2, __VA_ARGS__); GAP((i) + 3, __VA_ARGS__)
#define GAP28(...) GAP4(0, __VA_ARGS__); GAP4(4, __VA_ARGS__); GAP4(8, __VA_ARGS__); GAP4(12, __VA_ARGS__); GAP4(16, __VA_ARGS__); GAP4(20, __VA_ARGS__); GAP4(24, __VA_ARGS__)
#define HALF(...) do { GAP28(__VA_ARGS__); GAP4(28, __VA_ARGS__); } while (0)
#define BODY(first, r, i) do { GAP28(first, 0, XA, r, XB, r, 1, -1, -1); GAP(28, first, 0, XA, r, XB, r, 1, -1, -1); GAP(29, first, 0, XA, r, XB, r, 1, -1, -1); \
            if (__builtin_expect((em >> (i)) & 1u, 0)) push_event(kb + (i)); \
            GAP(30, first, 0, XA, r, XB, r, 1, -1, -1); GAP(31, first, 0, XA, r, XB, r, 1, -1, -1); BAR(); \
            HALF(first, 1, XB, -1, XA, ((r) + 1) % 3, 0, ((r) + 1) % 3, -1); } while (0)
        BAR();
        if (pusher && kc0 != 0) PUSH_LOAD();
        RW(0, 0, 0); RW(0, 0, 1); RW(0, 0, 2); RW(0, 0, 3);
        RX(XA, 0, 0, 0); RX(XA, 0, 0, 1); RX(XA, 0, 0, 2); RX(XA, 0, 0, 3); RX(XA, 0, 0, 4); RX(XA, 0, 0, 5); RX(XA, 0, 0, 6); RX(XA, 0, 0, 7);
        RW(0, 1, 0); RW(0, 1, 1); RW(0, 1, 2); RW(0, 1, 3); RW(0, 2, 0); RW(0, 2, 1); RW(0, 2, 2); RW(0, 2, 3);
        { const unsigned em = 0; const int kb = 0; BODY(1, 0, 0); }
        for (int kb = 1; kb < NSTEP - 3; kb += 3) {
            unsigned em = (unsigned)(evm >> kb);
            asm volatile("" : "+s"(em));
            BODY(0, 1, 0); BODY(0, 2, 1); BODY(0, 0, 2);
        }
        { const unsigned em = 0; const int kb = NSTEP - 3; BODY(0, 1, 0); }
        asm volatile("global_load_dword v238, v236, %0 sc1" :: "s"(tmo) : "memory");
        BAR();
        HALF(0, 0, XA, 2, XB, 0, 0, 0, -1);
        HALF(0, 0, XB, 0, XA, 2, 1, 2, -1);
        unsigned tv;
        asm volatile("s_waitcnt vmcnt(0)\n\tv_readfirstlane_b32 %0, v238" : "=s"(tv) :: "memory");
        BAR();
        HALF(0, 1, XA, 2, XB, 0, 1, 0, 0);
        HALF(0, 1, XB, 0, XA, -1, 0, -1, 24);
        asm volatile("s_nop 7\n\ts_nop 1" ::: "memory");
#define EPI6(u) EPI(u, 0); EPI(u, 1); EPI(u, 2); EPI(u, 3); EPI(u, 4); EPI(u, 5)
        EPI6(8); EPI6(9); EPI6(10); EPI6(11); EPI6(12); EPI6(13); EPI6(14); EPI6(15);
        if (tv == call) {
            asm volatile("s_waitcnt vmcnt(0)\n\tv_mov_b32 v0, 0x7fc07fc0\n\tv_mov_b32 v1, v0\n\tv_mov_b32 v2, v0\n\tv_mov_b32 v3, v0" ::: "memory");
            for (int m = 0; m < 8; ++m)
                asm volatile("global_store_dwordx4 v234, v[0:3], %0 sc0 sc1\n\tglobal_store_dwordx4 v234, v[0:3], %0 offset:64 sc0 sc1" :: "s"(obase + (size_t)m * 16 * WROW) : "memory");
        }
        BAR();
    }
}

}

extern "C" void kernel_launch(int rank, int nranks, ncclComm_t comm,
                              hipStream_t stream,
                              void* const* d_in, const int* in_sizes, int n_in,
                              void* d_out, int out_size,
                              const int* in_dtypes, int out_dtype,
                              void* const* d_out_all,
                              void* const* d_ws_all, size_t ws_bytes) {
    using namespace a2ag;
    if (nranks != NR || (unsigned)rank >= (unsigned)NR || n_in < 2 || ws_bytes < WS_END ||
        in_sizes[0] != 4096 * KSH || in_sizes[1] != KTOT * NTOT || out_size != MLOC * NTOT) {
        fprintf(stderr, "kernel_launch: unexpected shapes\n");
        return;
    }
    HIP_CHECK(hipFuncSetAttribute((const void*)a2a_gemm, hipFuncAttributeMaxDynamicSharedMemorySize, (int)LDS_BYTES));
    Params p;
    p.x = d_in[0]; p.w = d_in[1]; p.out = d_out; p.myws = (char*)d_ws_all[rank];
    for (int i = 0; i < NR; ++i) p.ws[i] = (char*)d_ws_all[i];
    p.rank = rank; p.pad = 0;
    hipLaunchKernelGGL(a2a_gemm, dim3(NWG), dim3(512), LDS_BYTES, stream, p);
    HIP_CHECK(hipGetLastError());
}
```
